```python
import numpy as np

kernel_name = "gpu_gemm_ar_m4096_k4096_n4096_bf16_gelu"


ABSMAX_RTOL = 0.008

COMPUTE_DTYPE = "bfloat16"
EPILOGUE = "gelu"
IN_QUANT = None
OUT_QUANT = None
IN_DTYPES = {"x": "bfloat16", "w_mat": "bfloat16"}
WORLD_SIZE = 8

def _to_bf16(x):
    u = np.ascontiguousarray(x, dtype=np.float32).view(np.uint32)
    return ((u + 0x7FFF + ((u >> 16) & 1)) & 0xFFFF0000).view(np.float32)

def setup_inputs():

    x = _to_bf16(np.random.randn(4096, 4096).astype(np.float32))
    w_mat = _to_bf16((np.random.randn(4096, 4096) / np.sqrt(4096)).astype(np.float32))
    return {"x": x, "w_mat": w_mat}

def _epilogue(y):
    c = 0.7978845608028654
    return (0.5 * y * (1.0 + np.tanh(c * (y + 0.044715 * y ** 3)))).astype(np.float32)

def reference(x, w_mat):
    y = (x @ w_mat).astype(np.float32)
    return _epilogue(y)
```

```cpp
#include <hip/hip_runtime.h>
#include <rccl/rccl.h>
#include <cstdio>
#include <cstdint>

#define HIP_CHECK(expr) do { hipError_t _e = (expr); if (_e != hipSuccess) { \
    fprintf(stderr, "HIP error %s at %s:%d\n", hipGetErrorString(_e), __FILE__, __LINE__); \
    return; } } while (0)

typedef unsigned u32;
typedef unsigned long long u64;
typedef unsigned u32x4 __attribute__((ext_vector_type(4)));
typedef unsigned u32x2 __attribute__((ext_vector_type(2)));
typedef short bf16x8 __attribute__((ext_vector_type(8)));
typedef float f32x4 __attribute__((ext_vector_type(4)));
typedef float f32x2 __attribute__((ext_vector_type(2)));
typedef __bf16 bf2 __attribute__((ext_vector_type(2)));
#define GAS __attribute__((address_space(1)))
#define LAS __attribute__((address_space(3)))
#define DEV __device__ __forceinline__

constexpr int NR = 8;
constexpr int NGEMM = 128;
#define PULL_PER_LINK 7
#define PULL_WAVES 4
constexpr int PPL = PULL_PER_LINK;
constexpr int WPW = PULL_WAVES;
constexpr int NPULL = 7 * PPL;
constexpr int GRID = 256;
#define PLACE 0
#define STAGGER 9
__host__ __device__ constexpr int role_of(int b) {
  return PLACE == 0 ? (b < 128 ? b : 128 + (255 - b))
       : PLACE == 1 ? ((b & 4) ? 128 : 0) + (b >> 3) * 4 + (b & 3)
       : ((b & 1) ? 128 : 0) + (b >> 1);
}
constexpr int NW = PPL * WPW;
constexpr u32 NIN = 384, NOUT = 512, NTOT = NIN + NOUT;
constexpr u64 DEADLINE = 100000000ull;

constexpr size_t OFF_CALL = 0;
constexpr size_t OFF_TMO = 131072;
constexpr size_t OFF_TKT = 131072 + 2048;
constexpr size_t OFF_DONE_A = 131072 + 4096;
constexpr size_t OFF_DONE_B = OFF_DONE_A + 8192;
constexpr size_t OFF_HIN = 262144;
constexpr size_t OFF_HOUT = OFF_HIN + 8 * NIN * 128;
constexpr size_t OFF_DBG = 1u << 20;
constexpr size_t OFF_CIN = 2u << 20;
constexpr size_t OFF_COUT = 10u << 20;
constexpr size_t OFF_AST = 16u << 20;
constexpr size_t OFF_BST = 32u << 20;
constexpr size_t WS_NEED = 40u << 20;
static_assert(OFF_HOUT + 8 * 128 * 128 <= OFF_DBG, "ctrl region");

constexpr int NSLOT = 4, IMG = 16384, SLOT = 32768;
constexpr int L_ENCU = 0;
constexpr int L_ENCS = 2048;
constexpr int L_DECU = 4096;
constexpr int L_DECS = 4352;
constexpr int L_XCH = 4608;
constexpr int L_TS = 5120;
constexpr int L_RING = 8192;
constexpr int L_VAR = L_RING + 65536;
constexpr int VARSZ = 4096 + 64;
constexpr int PBUF = 8192 + 64;
constexpr int LDS_BYTES = L_RING + 16 * PBUF;
static_assert(L_RING + NSLOT * SLOT <= LDS_BYTES && LDS_BYTES <= 163840, "lds");

struct Tab { unsigned short base[32]; unsigned char w[32]; };
constexpr Tab TU = {{0xec0, 0xf40, 0xf60, 0xf70, 0xf80, 0xf88, 0xf90, 0xf98, 0xfa0, 0xfa4, 0xfa8, 0xfb0, 0xfb8, 0xfc0, 0xfc4, 0xfc8,
                     0xfcc, 0xfd0, 0xfd4, 0xfd8, 0xfdc, 0xfe0, 0xfe2, 0xfe4, 0xfe6, 0xfea, 0xfee, 0xff2, 0xff6, 0xffe, 0x1006, 0xec0},
                    {7, 5, 4, 4, 3, 3, 3, 3, 2, 2, 3, 3, 3, 2, 2, 2, 2, 2, 2, 2, 2, 1, 1, 1, 2, 2, 2, 2, 3, 3, 4, 13}};
constexpr Tab TG = {{0x750, 0x790, 0x7b0, 0x7c0, 0x7c8, 0x7d0, 0x7d4, 0x7d8, 0x7dc, 0x7e0, 0x7e4, 0x7e8, 0x7ec, 0x7f0, 0x7f4, 0x7f8,
                     0x7fc, 0x1750, 0x1790, 0x17a0, 0x17a8, 0x17b0, 0x17b4, 0x17b8, 0x17bc, 0x17c0, 0x17c1, 0x17c2, 0x17c3, 0x17c4, 0x17c5, 0},
                    {6, 5, 4, 3, 3, 2, 2, 2, 2, 2, 2, 2, 2, 2, 2, 2, 4, 6, 4, 3, 3, 2, 2, 2, 2, 0, 0, 0, 0, 0, 0, 13}};
constexpr u32 U_U0 = 0xec0, S_U0 = 0x750;
struct LdsTabs { u32 t[1152]; };
constexpr LdsTabs make_tabs() {
  LdsTabs r{};
  for (u32 i = 0; i < 512; ++i) {
    u32 eu = 13u | (31u << 10), es = 13u | (31u << 11);
    const u32 us = (i >> 1) + S_U0 + ((i & 1) << 12);
    for (u32 c = 0; c < 31; ++c) {
      if (i - (TU.base[c] - U_U0) < (1u << TU.w[c])) eu = TU.w[c] | (c << 10) | ((TU.base[c] - U_U0) << 18);
      if (us - TG.base[c] < (1u << TG.w[c])) es = TG.w[c] | (c << 11) | ((u32)TG.base[c] << 19);
    }
    r.t[L_ENCU / 4 + i] = eu; r.t[L_ENCS / 4 + i] = es;
  }
  for (u32 c = 0; c < 32; ++c) {
    r.t[L_DECU / 4 + c] = ((u32)TU.base[c] << 18) | TU.w[c];
    r.t[L_DECS / 4 + 2 * c] = ((u32)TG.base[c] << 19) | TG.w[c];
  }
  return r;
}
__device__ const LdsTabs __attribute__((aligned(16))) k_tabs = make_tabs();
struct TileTab { u32 t[128]; };
constexpr TileTab make_tiles() {
  TileTab r{};
  u32 o = 0;
  for (u32 n = 0; n < 8; ++n) {
    for (u32 i = 0; i < n; ++i) r.t[o++] = 8 * i + n;
    for (u32 j = 0; j <= n; ++j) r.t[o++] = 8 * n + j;
  }
  for (; o < 128; ++o) r.t[o] = o;
  return r;
}
__device__ const TileTab k_tiles = make_tiles();
constexpr u32 INFO_RAW = 1u << 12, INFO_BAD = 1u << 13;

struct Args {
  const char* x;
  const char* w;
  char* out;
  char* ws[NR];
  int rank;
  int pad;
};

DEV LAS char* lds(u32 off) { return (LAS char*)(size_t)off; }
DEV u32 rfl(u32 v) { return __builtin_amdgcn_readfirstlane(v); }
DEV __amdgpu_buffer_rsrc_t mkrsrc(const void* p, u32 bytes) {
  return __builtin_amdgcn_make_buffer_rsrc((void*)p, 0, bytes, 0x00020000);
}
DEV void drain() { asm volatile("s_waitcnt vmcnt(0)" ::: "memory"); }
DEV void cfence() { asm volatile("" ::: "memory"); }
DEV u32 scan_incl(u32 x) {
  x += __builtin_amdgcn_update_dpp(0u, x, 0x111, 0xf, 0xf, false);
  x += __builtin_amdgcn_update_dpp(0u, x, 0x112, 0xf, 0xf, false);
  x += __builtin_amdgcn_update_dpp(0u, x, 0x114, 0xf, 0xf, false);
  x += __builtin_amdgcn_update_dpp(0u, x, 0x118, 0xf, 0xf, false);
  x += __builtin_amdgcn_update_dpp(0u, x, 0x142, 0xa, 0xf, false);
  x += __builtin_amdgcn_update_dpp(0u, x, 0x143, 0xc, 0xf, false);
  return x;
}
#define TS(slot) do {} while (0)
#define TS_INIT() do {} while (0)
#define TS_DUMP() do {} while (0)
DEV void wg_barrier() {
  __builtin_amdgcn_s_waitcnt(0xC07F);
  __builtin_amdgcn_s_barrier();
  cfence();
}

DEV void zero_var(int wid, int lane) {
  LAS char* var = (LAS char*)lds(L_VAR + (wid >> 1) * VARSZ + (wid & 1) * 2048);
  *(LAS u32x4*)(var + lane * 16) = u32x4{0, 0, 0, 0};
  *(LAS u32x4*)(var + 1024 + lane * 16) = u32x4{0, 0, 0, 0};
  if (lane < 4) *(LAS u32x4*)(var + 2048 + lane * 16) = u32x4{0, 0, 0, 0};
}
template <bool S>
DEV u32 encode_half(const u32 (&v)[16], int wid, int lane, char* dst) {
  const int h = wid & 1;
  LAS u32* var = (LAS u32*)lds(L_VAR + (wid >> 1) * VARSZ);
  LAS const char* enct = (LAS const char*)lds((S ? L_ENCS : L_ENCU));
  u32 delta = 0;
  if (!S) {
    u32 s = 0;
#pragma unroll
    for (int k = 0; k < 8; ++k) s += (v[2 * k] >> 7) & 0x00ff00ffu;
    s = (s & 0xffffu) + (s >> 16);
    s = __builtin_amdgcn_readlane(scan_incl(s), 63);
    delta = (u32)(((int)s - 128600 + 512) >> 10);
  }
  u32 shift = (U_U0 << 2) + (delta << 7);
  asm("" : "+s"(shift));
  u32 F[8], G[8], P[8], run = 0, mx = 0;
#pragma unroll
  for (int g = 0; g < 8; ++g) {
    const u32 A = v[2 * g], B = v[2 * g + 1];
    u32 e[4], o[4];
#pragma unroll
    for (int q = 0; q < 4; ++q) {
      const u32 x = (q & 1) ? ((q & 2 ? B : A) >> 16) : ((q & 2 ? B : A) & 0xffffu);
      if (S) {
        const u32 d = (x & 0x7ff8u) - (S_U0 << 3);
        const u32 ad = min(d, 255u * 8u) | ((x >> 13) & 4u);
        e[q] = *(LAS const u32*)(enct + ad);
        o[q] = __builtin_amdgcn_ubfe(x - (e[q] >> 16), 3, e[q]);
      } else {
        const u32 d = (x & 0x7ffcu) - shift;
        const u32 ad = min(d, 511u * 4u);
        e[q] = *(LAS const u32*)(enct + ad);
        o[q] = __builtin_amdgcn_ubfe(d - (e[q] >> 16), 2, e[q]);
      }
    }
    const u32 s01 = e[0] + e[1], s012 = s01 + e[2], W = (s012 + e[3]) & 0xffu;
    G[g] = o[0] | (o[1] << (e[0] & 31)) | (o[2] << (s01 & 31)) | (o[3] << (s012 & 31));
    P[g] = run; run += W; mx = max(mx, W);
    const u32 t02 = __builtin_amdgcn_perm(e[2], e[0], 0x0c0c0501u), t13 = __builtin_amdgcn_perm(e[3], e[1], 0x05010c0cu);
    if (S) F[g] = t02 | t13 | (A & 0x00070007u) | ((B & 0x00070007u) << 8);
    else { const u32 Bm = B & 0x80038003u; F[g] = t02 | t13 | (A & 0x80038003u) | __builtin_amdgcn_alignbit(Bm, Bm, 24); }
  }
  const u32 incl = scan_incl(run);
  const u32 mine = __builtin_amdgcn_readlane(incl, 63) | ((delta & 0xffu) << 16) | (__builtin_amdgcn_ballot_w64(mx > 32) != 0 ? 1u << 31 : 0u);
  if (lane == 0) *(LAS u32*)lds(L_XCH + wid * 4) = mine;
  wg_barrier();
  const u32 other = rfl(*(LAS const u32*)lds(L_XCH + (wid ^ 1) * 4));
  const u32 x0 = h ? other : mine, x1 = h ? mine : other;
  const u32 nvar = ((x0 & 0xffffu) + (x1 & 0xffffu) + 127) >> 7;
  const bool raw = ((x0 | x1) >> 31) || nvar >= 256;
  if (raw) {
    const __amdgpu_buffer_rsrc_t rd = mkrsrc(dst + h * 4096, 4096);
#pragma unroll
    for (int k = 0; k < 4; ++k)
      __builtin_amdgcn_raw_buffer_store_b128(u32x4{v[4 * k], v[4 * k + 1], v[4 * k + 2], v[4 * k + 3]}, rd, lane * 16 + k * 1024, 0, 17);
  } else {
    const __amdgpu_buffer_rsrc_t rd = mkrsrc(dst + h * 2048, 2048);
#pragma unroll
    for (int k = 0; k < 2; ++k)
      __builtin_amdgcn_raw_buffer_store_b128(u32x4{F[4 * k], F[4 * k + 1], F[4 * k + 2], F[4 * k + 3]}, rd, lane * 16 + k * 1024, 0, 17);
    const u32 base = incl - run + (h ? (x0 & 0xffffu) : 0u);
#pragma unroll
    for (int g = 0; g < 8; ++g) {
      const u32 pos = base + P[g];
      const u64 x = (u64)G[g] << (pos & 31);
      LAS u32* p = var + (pos >> 5);
      __hip_atomic_fetch_or(p, (u32)x, __ATOMIC_RELAXED, __HIP_MEMORY_SCOPE_WORKGROUP);
      __hip_atomic_fetch_or(p + 1, (u32)(x >> 32), __ATOMIC_RELAXED, __HIP_MEMORY_SCOPE_WORKGROUP);
    }
  }
  wg_barrier();
  if (raw) return INFO_RAW;
  const __amdgpu_buffer_rsrc_t rv = mkrsrc(dst + 4096, nvar * 16);
#pragma unroll 1
  for (u32 k = h; k * 64 < nvar; k += 2) {
    const u32x4 t = *(LAS const u32x4*)((LAS char*)var + k * 1024 + lane * 16);
    __builtin_amdgcn_raw_buffer_store_b128(t, rv, lane * 16 + k * 1024, 0, 17);
  }
  return nvar | (x0 & 0xff0000u) | ((x1 & 0xff0000u) << 8);
}

DEV u32 and_byte1(u32 x, u32 m) {
  u32 r;
  asm("v_and_b32_sdwa %0, %1, %2 dst_sel:DWORD dst_unused:UNUSED_PAD src0_sel:BYTE_1 src1_sel:DWORD" : "=v"(r) : "v"(x), "s"(m));
  return r;
}
DEV u32 pk_add16(u32 x, u32 y) {
  typedef unsigned short u16x2 __attribute__((ext_vector_type(2)));
  return __builtin_bit_cast(u32, (u16x2)(__builtin_bit_cast(u16x2, x) + __builtin_bit_cast(u16x2, y)));
}
template <bool S>
DEV void decode_block(u32 (&v)[32], u32 info, u32 lb, int lane) {
  LAS const char* buf = (LAS const char*)lds(lb);
  if (info & INFO_RAW) {
#pragma unroll
    for (int k = 0; k < 8; ++k) {
      const u32x4 t = *(LAS const u32x4*)(buf + k * 1024 + lane * 16);
      v[4 * k] = t[0]; v[4 * k + 1] = t[1]; v[4 * k + 2] = t[2]; v[4 * k + 3] = t[3];
    }
    return;
  }
  LAS const char* dect = (LAS const char*)lds((S ? L_DECS : L_DECU));
  u32 F[16];
#pragma unroll
  for (int k = 0; k < 4; ++k) {
    const u32x4 t = *(LAS const u32x4*)(buf + k * 1024 + lane * 16);
    F[4 * k] = t[0]; F[4 * k + 1] = t[1]; F[4 * k + 2] = t[2]; F[4 * k + 3] = t[3];
  }
  u32 e[16][4], s01[16], s012[16], P[16], run[2] = {0, 0};
#pragma unroll
  for (int g = 0; g < 16; ++g) {
    const u32 m = S ? 0xf8u : 0x7cu;
    e[g][0] = *(LAS const u32*)(dect + (F[g] & m));
    e[g][1] = *(LAS const u32*)(dect + ((F[g] >> 16) & m));
    e[g][2] = *(LAS const u32*)(dect + and_byte1(F[g], m));
    e[g][3] = *(LAS const u32*)(dect + ((F[g] >> 24) & m));
    s01[g] = e[g][0] + e[g][1]; s012[g] = s01[g] + e[g][2];
    P[g] = run[g >> 3]; run[g >> 3] += s012[g] + e[g][3];
  }
  const u32 both = __builtin_amdgcn_perm(run[1], run[0], 0x05040100u);
  const u32 incl = scan_incl(both), excl = incl - both;
  const u32 base[2] = {excl, (excl >> 16) + __builtin_amdgcn_readlane(incl, 63)};
  const u32 dsh[2] = {((info >> 16) & 0xffu) * 0x00800080u, (info >> 24) * 0x00800080u};
#pragma unroll
  for (int g = 0; g < 16; ++g) {
    const u32 pos = base[g >> 3] + P[g];
    LAS const u32* p = (LAS const u32*)(buf + 4096) + __builtin_amdgcn_ubfe(pos, 5, 11);
    const u32 win = __builtin_amdgcn_alignbit(p[1], p[0], pos);
    constexpr int SH = S ? 19 : 18;
    const u32 t0 = (__builtin_amdgcn_ubfe(win, 0, e[g][0]) << SH) + e[g][0];
    const u32 t1 = (__builtin_amdgcn_ubfe(win, e[g][0], e[g][1]) << SH) + e[g][1];
    const u32 t2 = (__builtin_amdgcn_ubfe(win, s01[g], e[g][2]) << SH) + e[g][2];
    const u32 t3 = (__builtin_amdgcn_ubfe(win, s012[g], e[g][3]) << SH) + e[g][3];
    u32 pa = __builtin_amdgcn_perm(t1, t0, 0x07060302u), pb = __builtin_amdgcn_perm(t3, t2, 0x07060302u);
    if (!S) { pa = pk_add16(pa, dsh[g >> 3]); pb = pk_add16(pb, dsh[g >> 3]); }
    const u32 m = S ? 0xfff8fff8u : 0x7ffc7ffcu;
    const u32 Fr = __builtin_amdgcn_alignbit(F[g], F[g], 8);
    v[2 * g] = (pa & m) | (F[g] & ~m);
    v[2 * g + 1] = (pb & m) | (Fr & ~m);
  }
}
DEV u32 block_bytes(u32 info) { return (info & INFO_RAW) ? 8192u : 4096u + (info & 0xffu) * 16u; }
DEV u32x4 srd(const char* p, u32 bytes) { return u32x4{(u32)(size_t)p, (u32)((size_t)p >> 32) & 0xffffu, bytes, 0x00020000u}; }
DEV void dma6(u32x4 rs, u32 lb, u32 vo0, u32 vo1) {
  u32 keep;
  asm volatile("s_nop 4\n\ts_mov_b32 %0, m0\n\ts_mov_b32 m0, %2\n\ts_nop 0\n\t"
               "buffer_load_dwordx4 %3, %1, 0 offen sc0 sc1 lds\n\t"
               "buffer_load_dwordx4 %3, %1, 0 offen offset:1024 sc0 sc1 lds\n\t"
               "buffer_load_dwordx4 %3, %1, 0 offen offset:2048 sc0 sc1 lds\n\t"
               "buffer_load_dwordx4 %3, %1, 0 offen offset:3072 sc0 sc1 lds\n\t"
               "s_add_u32 m0, %2, 0x1000\n\ts_nop 0\n\t"
               "buffer_load_dwordx4 %4, %1, 0 offen sc0 sc1 lds\n\t"
               "buffer_load_dwordx4 %4, %1, 0 offen offset:1024 sc0 sc1 lds\n\t"
               "s_mov_b32 m0, %0"
               : "=&s"(keep) : "s"(rs), "s"(lb), "v"(vo0), "v"(vo1) : "memory", "scc");
}
DEV void dma2_tail(u32x4 rs, u32 lb, u32 vo1) {
  u32 keep;
  asm volatile("s_nop 4\n\ts_mov_b32 %0, m0\n\ts_add_u32 m0, %2, 0x1000\n\ts_nop 0\n\t"
               "buffer_load_dwordx4 %3, %1, 0 offen offset:2048 sc0 sc1 lds\n\t"
               "buffer_load_dwordx4 %3, %1, 0 offen offset:3072 sc0 sc1 lds\n\t"
               "s_mov_b32 m0, %0"
               : "=&s"(keep) : "s"(rs), "s"(lb), "v"(vo1) : "memory", "scc");
}

DEV bool expired(u32* tmo, u32 call, u64 t0) {
  if (__hip_atomic_load(tmo, __ATOMIC_RELAXED, __HIP_MEMORY_SCOPE_AGENT) == call ||
      __builtin_amdgcn_s_memrealtime() - t0 > DEADLINE) {
    __hip_atomic_store(tmo, call, __ATOMIC_RELAXED, __HIP_MEMORY_SCOPE_AGENT);
    return true;
  }
  return false;
}
DEV bool wait_header(const u64* h, u32 call, u32* tmo, u32& info) {
  const u64 t0 = __builtin_amdgcn_s_memrealtime();
  for (;;) {
    const u64 x = __hip_atomic_load(h, __ATOMIC_RELAXED, __HIP_MEMORY_SCOPE_SYSTEM);
    const u32 c = rfl((u32)(x >> 32));
    info = rfl((u32)x);
    if (c >= call) return true;
    if (rfl(expired(tmo, call, t0))) return false;
    __builtin_amdgcn_s_sleep(2);
  }
}
DEV void wait_steps(const char* me, int pi, int pj, int lane, u64 need, u32 call, u32* tmo) {
  const u64* fa = (const u64*)(me + OFF_DONE_A) + pi * 64 + lane;
  const u64* fb = (const u64*)(me + OFF_DONE_B) + pj * 64 + lane;
  const u64 t0 = __builtin_amdgcn_s_memrealtime();
  for (;;) {
    const u64 x = __hip_atomic_load(fa, __ATOMIC_RELAXED, __HIP_MEMORY_SCOPE_AGENT);
    const u64 y = __hip_atomic_load(fb, __ATOMIC_RELAXED, __HIP_MEMORY_SCOPE_AGENT);
    const bool ok = (u32)x >= call && (u32)(x >> 32) >= call && (u32)y >= call && (u32)(y >> 32) >= call;
    if ((__builtin_amdgcn_ballot_w64(ok) & need) == need) return;
    if (rfl(expired(tmo, call, t0))) return;
    __builtin_amdgcn_s_sleep(4);
  }
}

DEV float gelu(float y) {
  const float k1 = -2.0f * 0.7978845608028654f * 1.4426950408889634f, k3 = k1 * 0.044715f;
  const float t = y * __builtin_fmaf(y * y, k3, k1);
  return y * __builtin_amdgcn_rcpf(1.0f + __builtin_amdgcn_exp2f(t));
}
DEV u32 pack_bf16(float lo, float hi) {
  return __builtin_bit_cast(u32, __builtin_convertvector(f32x2{lo, hi}, bf2));
}

__global__ void __launch_bounds__(512, 2) fused_kernel(Args a) {
  const int tid = threadIdx.x, lane = tid & 63, wid = rfl(tid >> 6), b = blockIdx.x;
  const int rank = a.rank, ri = rank >> 2, ci = rank & 3;
  char* const me = a.ws[rank];
  u32* const tmo = (u32*)(me + OFF_TMO);
  TS_INIT();
  TS(0);
  u32* const cnt = (u32*)(me + OFF_CALL + (b * 8 + wid) * 64);
  u32 call = __hip_atomic_load(cnt, __ATOMIC_RELAXED, __HIP_MEMORY_SCOPE_AGENT) + 1;

  {
    const int e = (wid >> 1) * GRID + b, h = wid & 1;
    int n, sub = e & 15, grp, t; bool isw = false;
    if (e < 768) { n = e / 96; const int r = e - 96 * n; sub = r & 15; isw = r < 64; grp = isw ? r >> 4 : (r - 64) >> 4; t = 32 * n + (isw ? 0 : 16) + sub; }
    else { n = 8 + ((e - 768) >> 5); grp = (e >> 4) & 1; t = 256 + 16 * (n - 8) + sub; }
    const int ks = sub >> 1, hf = sub & 1;
    u32 r[16], v[16];
    if (isw) {
      const char* src = a.w + (size_t)(64 * ks + 32 * hf + 8 * (lane >> 4)) * 8192 + (1024 * grp + 128 * n + 8 * (lane & 15) + 4 * h) * 2;
#pragma unroll
      for (int k = 0; k < 8; ++k) { const u32x2 q = *(const u32x2*)(src + k * 8192); r[2 * k] = q[0]; r[2 * k + 1] = q[1]; }
    } else {
      const int rr = lane >> 2;
      const char* src = a.x + (size_t)(2048 * grp + 128 * n + 64 * hf + 32 * h + rr) * 1024 + (64 * ks + 8 * ((lane & 3) ^ (2 * (rr >> 3)))) * 2;
#pragma unroll
      for (int k = 0; k < 4; ++k) {
        const u32x4 q = *(const u32x4*)(src + (k >> 1) * 16384 + (k & 1) * 64);
        r[4 * k] = q[0]; r[4 * k + 1] = q[1]; r[4 * k + 2] = q[2]; r[4 * k + 3] = q[3];
      }
    }
    if (tid < 288) *(LAS u32x4*)lds(tid * 16) = *(const u32x4*)&k_tabs.t[tid * 4];
    zero_var(wid, lane);
    wg_barrier();
    call = rfl(call);
    if (lane == 0) __hip_atomic_store(cnt, call, __ATOMIC_RELAXED, __HIP_MEMORY_SCOPE_AGENT);
    if (isw) {
#pragma unroll
      for (int c = 0; c < 4; ++c)
#pragma unroll
        for (int j = 0; j < 4; ++j)
          v[4 * c + j] = __builtin_amdgcn_perm(r[2 * (2 * j + 1) + (c >> 1)], r[2 * (2 * j) + (c >> 1)], (c & 1) ? 0x07060302u : 0x05040100u);
    } else {
#pragma unroll
      for (int k = 0; k < 16; ++k) v[k] = r[k];
    }
    TS(1);
    const bool self = isw ? (grp == ci) : (grp == ri);
    const int stepi = (n * 64 + rank * 8 + ks) * 2 + hf;
    const u32 info = encode_half<false>(v, wid, lane, me + OFF_CIN + (size_t)e * 8192);
    if (self) {
      const __amdgpu_buffer_rsrc_t rd = mkrsrc(me + (isw ? OFF_BST : OFF_AST) + (size_t)stepi * 8192 + h * 4096, 4096);
#pragma unroll
      for (int k = 0; k < 4; ++k)
        __builtin_amdgcn_raw_buffer_store_b128(u32x4{v[4 * k], v[4 * k + 1], v[4 * k + 2], v[4 * k + 3]}, rd, lane * 16 + k * 1024, 0, 17);
    }
    TS(2);
    drain();
    wg_barrier();
    if (h == 0) {
      const int d = isw ? grp + 4 * lane : 4 * grp + lane;
      if (lane < (isw ? 2 : 4) && d != rank && (unsigned)d < (unsigned)NR)
        __hip_atomic_store((u64*)(a.ws[d] + OFF_HIN + (rank * NIN + t) * 128), ((u64)call << 32) | info, __ATOMIC_RELAXED, __HIP_MEMORY_SCOPE_SYSTEM);
      if (self && lane == 0)
        __hip_atomic_store((u32*)(me + (isw ? OFF_DONE_B : OFF_DONE_A)) + stepi, call, __ATOMIC_RELAXED, __HIP_MEMORY_SCOPE_AGENT);
    }
    TS(3);
  }

  const int role = role_of(b);
  if (role >= NGEMM + NPULL) { TS_DUMP(); return; }
  if (role >= NGEMM) {
    if (wid >= WPW) { TS_DUMP(); return; }
    const int pb = role - NGEMM, link = pb % 7;
    const int p = (rank + 1 + link) & 7;
    const char* const peer = a.ws[p];
    const char* const tkp = me + OFF_TKT + link * 128;
    const u32 tkbase = (call - 1) * NTOT - 2 * NW;
    const u32 vo0 = lane * 16, vo1 = lane * 16 + 4096, voo = (lane >> 4) * 8192 + (lane & 15) * 16, zero = 0, one = 1;
    const u32 lb = L_RING + wid * PBUF;
    auto hdr_of = [&](u32 t) -> const char* {
      t = min(t, NTOT - 1);
      const u32 oi = (u32)OFF_HIN + (p * NIN + t) * 128, oo = (u32)OFF_HOUT + (p * 128 + ((t - NIN) >> 2)) * 128 + (t & 3) * 8;
      return me + (t < NIN ? oi : oo);
    };
    auto fetch = [&](u32 t, u32 info) {
      const u32 e0 = 96 * (t >> 5) + ((t & 16) ? 64 + 16 * ri : 16 * ci), e1 = 768 + 2 * ((t - 256) & ~15u) + 16 * ri;
      const u32 oi = (u32)OFF_CIN + ((t < 256 ? e0 : e1) + (t & 15)) * 8192, oo = (u32)OFF_COUT + (t - NIN) * 8192;
      const u32 nb = block_bytes(info);
      const u32x4 rs = srd(peer + (t < NIN ? oi : oo), nb);
      dma6(rs, lb, vo0, vo1);
      if (nb > 6144) dma2_tail(rs, lb, vo1);
    };
    auto fetch_wait = [&](u32 t) -> u32 {
      u32 info;
      if (!wait_header((const u64*)hdr_of(t), call, tmo, info)) return INFO_BAD;
      fetch(t, info);
      return info;
    };
    u32 T0 = (pb / 7) * WPW + wid, T1 = T0 + NW;
    u32 info0;
    if (!wait_header((const u64*)hdr_of(T0), call, tmo, info0)) info0 = INFO_BAD;
    else {
      const u64 ts = __builtin_amdgcn_s_memrealtime();
      while (__builtin_amdgcn_s_memrealtime() - ts < (u64)(T0 * STAGGER)) __builtin_amdgcn_s_sleep(1);
      fetch(T0, info0);
    }
    u32* pend = nullptr;
    u32 v[32];
    while (T0 < NTOT) {
      drain();
      if (pend && lane == 0) __hip_atomic_store(pend, call, __ATOMIC_RELAXED, __HIP_MEMORY_SCOPE_AGENT);
      u32 tk; u32x2 hd;
      asm volatile("s_nop 4\n\ts_mov_b64 exec, 1\n\tglobal_atomic_add %0, %2, %3, %4 sc0\n\ts_mov_b64 exec, -1\n\t"
                   "global_load_dwordx2 %1, %2, %5 sc0 sc1"
                   : "=&v"(tk), "=&v"(hd) : "v"(zero), "v"(one), "s"(tkp), "s"(hdr_of(T1)) : "memory");
      if (info0 == INFO_BAD) {
#pragma unroll
        for (int k = 0; k < 32; ++k) v[k] = 0x7fc07fc0u;
      } else if (T0 < NIN) decode_block<false>(v, info0, lb, lane);
      else decode_block<true>(v, info0, lb, lane);
      u32 tk2; u32x2 hd2;
      asm volatile("s_waitcnt vmcnt(0)\n\tv_mov_b32 %0, %2\n\tv_mov_b64 %1, %3" : "=&v"(tk2), "=&v"(hd2) : "v"(tk), "v"(hd) : "memory");
      const u32 T2 = rfl(tk2) - tkbase;
      u32 info1 = rfl(hd2[0]);
      const bool pre = rfl(hd2[1]) >= call;
      if (pre && T1 < NTOT) fetch(T1, info1);
      const bool isin = T0 < NIN, isw = T0 < 256 && !(T0 & 16);
      const u32 stepi = (T0 < 256 ? T0 >> 5 : 8 + ((T0 - 256) >> 4)) * 128 + p * 16 + (T0 & 15);
      const u32 tl = k_tiles.t[((T0 >> 2) - NIN / 4) & 127], ti = tl >> 3, tj = tl & 7;
      const u32 oi = (u32)(isw ? OFF_BST : OFF_AST) + stepi * 8192;
      const u32 oo = (2048 * (p >> 2) + 128 * ti + 32 * (T0 & 3)) * 8192 + (1024 * (p & 3) + 128 * tj) * 2;
      pend = isin ? (u32*)(me + (isw ? OFF_DONE_B : OFF_DONE_A)) + stepi : nullptr;
      const __amdgpu_buffer_rsrc_t rd = mkrsrc((isin ? me : a.out) + rfl(isin ? oi : oo), isin ? 8192 : 31 * 8192 + 256);
      const u32 vo = isin ? vo0 : voo, vs = isin ? 1024 : 32768;
#pragma unroll
      for (int k = 0; k < 8; ++k)
        __builtin_amdgcn_raw_buffer_store_b128(u32x4{v[4 * k], v[4 * k + 1], v[4 * k + 2], v[4 * k + 3]}, rd, vo + k * vs, 0, 17);
      if (!pre && T1 < NTOT) {
        drain();
        if (pend && lane == 0) __hip_atomic_store(pend, call, __ATOMIC_RELAXED, __HIP_MEMORY_SCOPE_AGENT);
        pend = nullptr;
        info1 = fetch_wait(T1);
      }
      T0 = T1; info0 = info1; T1 = T2;
    }
    drain();
    if (pend && lane == 0) __hip_atomic_store(pend, call, __ATOMIC_RELAXED, __HIP_MEMORY_SCOPE_AGENT);
    TS(31);
    TS_DUMP();
    return;
  }

  const int pi = k_tiles.t[role] >> 3, pj = k_tiles.t[role] & 7;
  if (wid == 0) wait_steps(me, pi, pj, lane, 0xffull << (8 * rank), call, tmo);
  wg_barrier();
  TS(4);
  if (wid >= 4) {
    const int lw = wid - 4;
    const char* ga = me + OFF_AST + (size_t)pi * 64 * IMG + lw * 4096 + lane * 16;
    const char* gb = me + OFF_BST + (size_t)pj * 64 * IMG + lw * 4096 + lane * 16;
    auto issue = [&](int s, int slot) {
      const int st = (8 * rank + s) & 63;
      LAS char* la = lds(L_RING + slot * SLOT + lw * 4096);
#pragma unroll
      for (int i = 0; i < 4; ++i)
        __builtin_amdgcn_global_load_lds((const GAS void*)(ga + (size_t)st * IMG + i * 1024), (LAS void*)(la + i * 1024), 16, 0, 17);
#pragma unroll
      for (int i = 0; i < 4; ++i)
        __builtin_amdgcn_global_load_lds((const GAS void*)(gb + (size_t)st * IMG + i * 1024), (LAS void*)(la + IMG + i * 1024), 16, 0, 17);
    };
    for (int s = 0; s < NSLOT; ++s) issue(s, s);
    asm volatile("s_waitcnt vmcnt(24)" ::: "memory");
    __builtin_amdgcn_s_barrier();
    asm volatile("s_waitcnt vmcnt(16)" ::: "memory");
    __builtin_amdgcn_s_barrier();
    for (int s = 0; s < 64; ++s) {
      if (s + NSLOT < 64) {
        issue(s + NSLOT, s & (NSLOT - 1));
        asm volatile("s_waitcnt vmcnt(16)" ::: "memory");
      } else {
        asm volatile("s_waitcnt vmcnt(0)" ::: "memory");
      }
      __builtin_amdgcn_s_barrier();
    }
  } else {
    const int wr = wid >> 1, wc = wid & 1;
    const int row = lane & 15, c = lane >> 4;
    const int aoff = L_RING + wr * 8192 + 16 * (4 * row + (c ^ (2 * (row >> 3))));
    const int boff = L_RING + IMG + wc * 4096 + 16 * lane;
    f32x4 acc[4][4];
#pragma unroll
    for (int i = 0; i < 4; ++i)
#pragma unroll
      for (int j = 0; j < 4; ++j) acc[i][j] = f32x4{0, 0, 0, 0};
    bf16x8 a0[2][4], b0[2][4], a1[2][4], b1[2][4];
#define RD(A_, B_, slot_) do { LAS const char* pa = lds((slot_) * SLOT + aoff); LAS const char* pb = lds((slot_) * SLOT + boff); \
    _Pragma("unroll") for (int k = 0; k < 2; ++k) { \
      _Pragma("unroll") for (int i = 0; i < 4; ++i) A_[k][i] = *(LAS const bf16x8*)(pa + i * 2048 + k * 1024); \
      _Pragma("unroll") for (int i = 0; i < 4; ++i) B_[k][i] = *(LAS const bf16x8*)(pb + k * 8192 + i * 1024); } } while (0)
#define MM(A_, B_) do { _Pragma("unroll") for (int k = 0; k < 2; ++k) \
    _Pragma("unroll") for (int i = 0; i < 4; ++i) _Pragma("unroll") for (int j = 0; j < 4; ++j) \
      acc[i][j] = __builtin_amdgcn_mfma_f32_16x16x32_bf16(A_[k][i], B_[k][j], acc[i][j], 0, 0, 0); } while (0)
#define SCHEDG do { _Pragma("unroll") for (int q = 0; q < 16; ++q) { __builtin_amdgcn_sched_group_barrier(0x8, 1, 0); \
    __builtin_amdgcn_sched_group_barrier(0x100, 1, 0); } __builtin_amdgcn_sched_group_barrier(0x8, 16, 0); } while (0)
    __builtin_amdgcn_s_barrier();
    RD(a0, b0, 0);
    __builtin_amdgcn_s_waitcnt(0xC07F);
    __builtin_amdgcn_sched_barrier(0);
    __builtin_amdgcn_s_barrier();
    for (int s = 0; s < 64; s += 2) {
      RD(a1, b1, (s + 1) & (NSLOT - 1)); MM(a0, b0); SCHEDG;
      __builtin_amdgcn_sched_barrier(0);
      __builtin_amdgcn_s_waitcnt(0xC07F);
      __builtin_amdgcn_s_barrier();
      __builtin_amdgcn_sched_barrier(0);
      RD(a0, b0, (s + 2) & (NSLOT - 1)); MM(a1, b1); SCHEDG;
      __builtin_amdgcn_sched_barrier(0);
      __builtin_amdgcn_s_waitcnt(0xC07F);
      if (s == 8 - NSLOT - 2 && wid == 0) { wait_steps(me, pi, pj, lane, ~0ull, call, tmo); TS(5); }
      __builtin_amdgcn_s_barrier();
      __builtin_amdgcn_sched_barrier(0);
    }
    TS(6);
#pragma unroll
    for (int i = 0; i < 4; ++i)
#pragma unroll
      for (int r = 0; r < 4; ++r)
        *(LAS f32x4*)lds(L_RING + wc * 32768 + (64 * wr + 16 * i + 4 * c + r) * 256 + 16 * row) =
            f32x4{acc[i][0][r], acc[i][1][r], acc[i][2][r], acc[i][3][r]};
  }
  const bool bad = __hip_atomic_load(tmo, __ATOMIC_RELAXED, __HIP_MEMORY_SCOPE_AGENT) == call;
  zero_var(wid, lane);
  wg_barrier();
  TS(7);
  {
    u32 v[16];
#pragma unroll
    for (int k = 0; k < 4; ++k) {
      const f32x4 y0 = *(LAS const f32x4*)lds(L_RING + wid * 4096 + k * 1024 + lane * 16);
      const f32x4 y1 = *(LAS const f32x4*)lds(L_RING + 32768 + wid * 4096 + k * 1024 + lane * 16);
      v[4 * k] = pack_bf16(gelu(y0[0]), gelu(y0[1])); v[4 * k + 1] = pack_bf16(gelu(y0[2]), gelu(y0[3]));
      v[4 * k + 2] = pack_bf16(gelu(y1[0]), gelu(y1[1])); v[4 * k + 3] = pack_bf16(gelu(y1[2]), gelu(y1[3]));
    }
    if (bad) {
#pragma unroll
      for (int k = 0; k < 16; ++k) v[k] = 0x7fc07fc0u;
    }
    const u32 info = encode_half<true>(v, wid, lane, me + OFF_COUT + (size_t)(role * 4 + (wid >> 1)) * 8192);
    if (lane == 0) *(LAS u32*)lds(L_XCH + 32 + wid * 4) = info;
    TS(8);
    drain();
    wg_barrier();
    if (wid == 0 && lane < 28) {
      const int d = (rank + 1 + (lane >> 2)) & 7;
      const u32 inf = *(LAS const u32*)lds(L_XCH + 32 + (lane & 3) * 8);
      __hip_atomic_store((u64*)(a.ws[d] + OFF_HOUT + (rank * 128 + role) * 128 + (lane & 3) * 8), ((u64)call << 32) | inf,
                         __ATOMIC_RELAXED, __HIP_MEMORY_SCOPE_SYSTEM);
    }
    TS(9);
    const __amdgpu_buffer_rsrc_t rd = mkrsrc(a.out + (size_t)(2048 * ri + 128 * pi + 16 * wid) * 8192 + (1024 * ci + 128 * pj) * 2, 15 * 8192 + 256);
#pragma unroll
    for (int k = 0; k < 4; ++k)
      __builtin_amdgcn_raw_buffer_store_b128(u32x4{v[4 * k], v[4 * k + 1], v[4 * k + 2], v[4 * k + 3]}, rd,
                                             (lane >> 4) * 8192 + (lane & 15) * 16 + k * 32768, 0, 17);
  }
  TS_DUMP();
}

extern "C" void kernel_launch(int rank, int nranks, ncclComm_t comm,
                              hipStream_t stream,
                              void* const* d_in, const int* in_sizes, int n_in,
                              void* d_out, int out_size,
                              const int* in_dtypes, int out_dtype,
                              void* const* d_out_all,
                              void* const* d_ws_all, size_t ws_bytes) {
  (void)comm; (void)in_dtypes; (void)out_dtype; (void)d_out_all;
  if (nranks != NR || (unsigned)rank >= (unsigned)NR || n_in < 2 || ws_bytes < WS_NEED ||
      in_sizes[0] != 4096 * 512 || in_sizes[1] != 512 * 4096 || out_size != 4096 * 4096) {
    fprintf(stderr, "kernel_launch: unexpected problem shape\n");
    return;
  }
  HIP_CHECK(hipFuncSetAttribute((const void*)fused_kernel, hipFuncAttributeMaxDynamicSharedMemorySize, LDS_BYTES));
  Args a;
  a.x = (const char*)d_in[0];
  a.w = (const char*)d_in[1];
  a.out = (char*)d_out;
  for (int p = 0; p < NR; ++p) a.ws[p] = (char*)d_ws_all[p];
  a.rank = rank;
  a.pad = 0;
  fused_kernel<<<GRID, 512, LDS_BYTES, stream>>>(a);
  HIP_CHECK(hipGetLastError());
}
```

```cpp
#include <hip/hip_runtime.h>
#include <rccl/rccl.h>
#include <cstdio>
#include <cstdint>

#define HIP_CHECK(expr) do { hipError_t _e = (expr); if (_e != hipSuccess) { \
    fprintf(stderr, "HIP error %s at %s:%d\n", hipGetErrorString(_e), __FILE__, __LINE__); \
    return; } } while (0)

typedef unsigned u32;
typedef unsigned long long u64;
typedef unsigned u32x4 __attribute__((ext_vector_type(4)));
typedef unsigned u32x2 __attribute__((ext_vector_type(2)));
typedef short bf16x8 __attribute__((ext_vector_type(8)));
typedef float f32x4 __attribute__((ext_vector_type(4)));
typedef float f32x2 __attribute__((ext_vector_type(2)));
typedef __bf16 bf2 __attribute__((ext_vector_type(2)));
#define GAS __attribute__((address_space(1)))
#define LAS __attribute__((address_space(3)))
#define DEV __device__ __forceinline__

constexpr int NR = 8;
#define PULL_PER_LINK 8
#define LINK_GBS 61.0
#define HLOOK 4
#define EARLIEST 800
#define PSLEEP 8
#define RAWHEAD 16
#define PUSH_T0 190
#define PUSH_DT 15
#define PULL_T0 0
#define HDR_T0 400
#define HDR_DT8 8
#define KSTEP 4
#define LOOK 6
#define SLOTB 9472
#define RAWTAIL 3
constexpr int PPL = PULL_PER_LINK;
constexpr int NPULL = 7 * PPL;
constexpr int GRID = 256;
#define HALF_LO 0
#define HALF_HI 64
#define GROUP 8
static_assert(128 + HALF_HI - HALF_LO + NPULL <= GRID && NPULL <= 64, "roles");
constexpr u32 TLINE = (u32)(PPL * 128.0 / LINK_GBS * 25.6 + 0.5);
constexpr u32 NIN = 384, NOUT = 512, NTOT = NIN + NOUT;
constexpr u64 DEADLINE = 100000000ull;

constexpr size_t OFF_CALL = 0;
constexpr size_t OFF_TMO = 131072;
constexpr size_t OFF_TKT = 131072 + 2048;
constexpr size_t OFF_DONE_A = 131072 + 4096;
constexpr size_t OFF_DONE_B = OFF_DONE_A + 8192;
constexpr size_t OFF_HIN = 262144;
constexpr size_t OFF_HOUT = OFF_HIN + 8 * NIN * 128;
constexpr size_t OFF_DBG = 1u << 20;
constexpr size_t OFF_CIN = 40u << 20;
constexpr size_t OFF_COUT = 6u << 20;
constexpr size_t OFF_AST = 16u << 20;
constexpr size_t OFF_BST = 32u << 20;
constexpr size_t WS_NEED = 56u << 20;
static_assert(SLOTB >= 8192 && SLOTB <= 16384 && SLOTB % 128 == 0, "slot");
static_assert(OFF_HOUT + 8 * 128 * 128 <= OFF_DBG, "ctrl region");

constexpr int NSLOT = 4, IMG = 16384, SLOT = 32768;
constexpr int L_ENCU = 0;
constexpr int L_ENCS = 2048;
constexpr int L_DECU = 4096;
constexpr int L_DECS = 4352;
constexpr int L_XCH = 4608;
constexpr int L_SYNC = 4864;
constexpr int L_PQ = 4640;
constexpr u32 NBUF = 16;
constexpr int L_TS = 5120;
constexpr int L_RING = 8192;
constexpr int VARSZ = 4096 + 64;
constexpr int PBUF = 8192 + 64;
constexpr int L_VAR = L_RING + 16 * PBUF + 2048;
constexpr int LDS_BYTES = L_VAR + 4 * VARSZ;
static_assert(L_RING + NSLOT * SLOT <= LDS_BYTES && LDS_BYTES <= 163840, "lds");

struct Tab { unsigned short base[32]; unsigned char w[32]; };
constexpr Tab TU = {{0xec0, 0xf40, 0xf60, 0xf70, 0xf80, 0xf88, 0xf90, 0xf98, 0xfa0, 0xfa4, 0xfa8, 0xfb0, 0xfb8, 0xfc0, 0xfc4, 0xfc8,
                     0xfcc, 0xfd0, 0xfd4, 0xfd8, 0xfdc, 0xfe0, 0xfe2, 0xfe4, 0xfe6, 0xfea, 0xfee, 0xff2, 0xff6, 0xffe, 0x1006, 0xec0},
                    {7, 5, 4, 4, 3, 3, 3, 3, 2, 2, 3, 3, 3, 2, 2, 2, 2, 2, 2, 2, 2, 1, 1, 1, 2, 2, 2, 2, 3, 3, 4, 13}};
constexpr Tab TG = {{0x750, 0x790, 0x7b0, 0x7c0, 0x7c8, 0x7d0, 0x7d4, 0x7d8, 0x7dc, 0x7e0, 0x7e4, 0x7e8, 0x7ec, 0x7f0, 0x7f4, 0x7f8,
                     0x7fc, 0x1750, 0x1790, 0x17a0, 0x17a8, 0x17b0, 0x17b4, 0x17b8, 0x17bc, 0x17c0, 0x17c1, 0x17c2, 0x17c3, 0x17c4, 0x17c5, 0},
                    {6, 5, 4, 3, 3, 2, 2, 2, 2, 2, 2, 2, 2, 2, 2, 2, 4, 6, 4, 3, 3, 2, 2, 2, 2, 0, 0, 0, 0, 0, 0, 13}};
constexpr u32 U_U0 = 0xec0, S_U0 = 0x750;
struct LdsTabs { u32 t[1152]; };
constexpr LdsTabs make_tabs() {
  LdsTabs r{};
  for (u32 i = 0; i < 512; ++i) {
    u32 eu = 13u | (31u << 10), es = 13u | (31u << 11);
    const u32 us = (i >> 1) + S_U0 + ((i & 1) << 12);
    for (u32 c = 0; c < 31; ++c) {
      if (i - (TU.base[c] - U_U0) < (1u << TU.w[c])) eu = TU.w[c] | (c << 10) | ((TU.base[c] - U_U0) << 18);
      if (us - TG.base[c] < (1u << TG.w[c])) es = TG.w[c] | (c << 11) | ((u32)TG.base[c] << 19);
    }
    r.t[L_ENCU / 4 + i] = eu; r.t[L_ENCS / 4 + i] = es;
  }
  for (u32 c = 0; c < 32; ++c) {
    r.t[L_DECU / 4 + c] = ((u32)TU.base[c] << 18) | TU.w[c];
    r.t[L_DECS / 4 + 2 * c] = ((u32)TG.base[c] << 19) | TG.w[c];
  }
  return r;
}
__device__ const LdsTabs __attribute__((aligned(16))) k_tabs = make_tabs();
struct TileTab { u32 t[128]; };
constexpr TileTab make_tiles() {
  TileTab r{};
  u32 o = 0;
  for (u32 n = 0; n < 8; ++n) {
    for (u32 i = 0; i < n; ++i) r.t[o++] = 8 * i + n;
    for (u32 j = 0; j <= n; ++j) r.t[o++] = 8 * n + j;
  }
  for (; o < 128; ++o) r.t[o] = o;
  return r;
}
__device__ const TileTab k_tiles = make_tiles();
constexpr u32 R_PULL = 0x1000, R_IDLE = 0x2000;
#define ISO 1
__host__ __device__ constexpr bool is_pull(u32 b) { return ISO ? !(b & 1) && b >= 256 - 2 * NPULL : b >= 192; }
struct RoleTab { u32 t[256]; };
constexpr RoleTab make_roles() {
  RoleTab r{};
  u32 used[8] = {};
  if (ISO) {
    u32 job = 0;
    for (u32 run = 1; run < 16; ++run) {
      u32 c = 0;
      for (u32 k = 1; k < 4; ++k) if (used[k] < used[c]) c = k;
      for (u32 t = 0; t < (run + 1) / 2; ++t) {
        if (used[c] > 30) for (u32 k = 0; k < 4; ++k) if (used[k] < used[c]) c = k;
        for (u32 h = 0; h < 2; ++h, ++job, ++used[c]) {
          const u32 j2 = 128 + job;
          r.t[8 * used[c] + 2 * c + 1] = ((job >> 1) | (0x100 << (job & 1))) | ((j2 >> 1) | (0x100 << (j2 & 1))) << 16;
        }
      }
    }
    for (u32 b = 0; b < 256; b += 2) r.t[b] = is_pull(b) ? R_PULL + (b - (256 - 2 * NPULL)) / 2 : R_IDLE;
    return r;
  }
  for (u32 b = 0; b < 256; ++b) r.t[b] = b >= 192 && b < 192 + NPULL ? R_PULL + b - 192 : R_IDLE;
  auto room = [&](u32 x) { return 24 + (192 + NPULL + 7 - x < 256 ? (256 - 192 - NPULL + x) / 8 : 0) - used[x]; };
  auto put = [&](u32 x, u32 v) { const u32 k = used[x]++; r.t[k < 24 ? 8 * k + x : 8 * (31 - (k - 24)) + x] = v; };
  u32 o = 0;
  for (u32 run = 0; run < 24; ++run) {
    const u32 cnt = run < 16 ? (run + 1) / 2 : 8;
    u32 x = 0;
    for (u32 t = 0; t < cnt; ++t, ++o) {
      const u32 need = o >= HALF_LO && o < HALF_HI ? 2 : 1;
      if (t % GROUP == 0 || room(x) < need)
        for (u32 k = 0; k < 8; ++k) if (room(k) > room(x)) x = k;
      if (need == 2) { put(x, o | 0x100); if (room(x) < 1) for (u32 k = 0; k < 8; ++k) if (room(k) > room(x)) x = k; put(x, o | 0x200); }
      else put(x, o);
    }
  }
  return r;
}
constexpr RoleTab h_roles = make_roles();
__device__ const RoleTab k_roles = make_roles();
constexpr u32 INFO_RAW = 1u << 12, INFO_BAD = 1u << 13, INFO_END = 1u << 14;

struct Args {
  const char* x;
  const char* w;
  char* out;
  char* ws[NR];
  int rank;
  int pad;
};

DEV LAS char* lds(u32 off) { return (LAS char*)(size_t)off; }
DEV u32 rfl(u32 v) { return __builtin_amdgcn_readfirstlane(v); }
DEV __amdgpu_buffer_rsrc_t mkrsrc(const void* p, u32 bytes) {
  return __builtin_amdgcn_make_buffer_rsrc((void*)p, 0, bytes, 0x00020000);
}
DEV void drain() { asm volatile("s_waitcnt vmcnt(0)" ::: "memory"); }
DEV void cfence() { asm volatile("" ::: "memory"); }
DEV u32 scan_incl(u32 x) {
  x += __builtin_amdgcn_update_dpp(0u, x, 0x111, 0xf, 0xf, false);
  x += __builtin_amdgcn_update_dpp(0u, x, 0x112, 0xf, 0xf, false);
  x += __builtin_amdgcn_update_dpp(0u, x, 0x114, 0xf, 0xf, false);
  x += __builtin_amdgcn_update_dpp(0u, x, 0x118, 0xf, 0xf, false);
  x += __builtin_amdgcn_update_dpp(0u, x, 0x142, 0xa, 0xf, false);
  x += __builtin_amdgcn_update_dpp(0u, x, 0x143, 0xc, 0xf, false);
  return x;
}
#define TS(slot) do {} while (0)
#define TS_INIT() do {} while (0)
#define TS_DUMP() do {} while (0)
DEV void wg_barrier() {
  __builtin_amdgcn_s_waitcnt(0xC07F);
  __builtin_amdgcn_s_barrier();
  cfence();
}

DEV u32 lds_get(u32 off) { cfence(); const u32 x = rfl(*(volatile LAS u32*)lds(off)); cfence(); return x; }
DEV void lds_put(u32 off, u32 x) { cfence(); *(volatile LAS u32*)lds(off) = x; cfence(); }
DEV u32 pair_swap(int wid, int k, u32 x) {
  lds_put(L_SYNC + wid * 32 + k * 4, x);
  u32 y;
  while ((y = lds_get(L_SYNC + (wid ^ 1) * 32 + k * 4)) == 0) {}
  return y;
}

DEV void zero_var(int wid, int lane) {
  LAS char* var = (LAS char*)lds(L_VAR + (wid >> 1) * VARSZ + (wid & 1) * 2048);
  *(LAS u32x4*)(var + lane * 16) = u32x4{0, 0, 0, 0};
  *(LAS u32x4*)(var + 1024 + lane * 16) = u32x4{0, 0, 0, 0};
  if (lane < 4) *(LAS u32x4*)(var + 2048 + lane * 16) = u32x4{0, 0, 0, 0};
}
template <bool S>
DEV u32 encode_half(const u32 (&v)[16], int wid, int lane, char* dst, bool force_raw, int sk) {
  const int h = wid & 1;
  LAS u32* var = (LAS u32*)lds(L_VAR + (wid >> 1) * VARSZ);
  LAS const char* enct = (LAS const char*)lds((S ? L_ENCS : L_ENCU));
  u32 delta = 0;
  if (!S) {
    u32 s = 0;
#pragma unroll
    for (int k = 0; k < 8; ++k) s += (v[2 * k] >> 7) & 0x00ff00ffu;
    s = (s & 0xffffu) + (s >> 16);
    s = __builtin_amdgcn_readlane(scan_incl(s), 63);
    delta = (u32)(((int)s - 128600 + 512) >> 10);
  }
  u32 shift = (U_U0 << 2) + (delta << 7);
  asm("" : "+s"(shift));
  u32 F[8], G[8], P[8], run = 0, mx = 0;
#pragma unroll
  for (int g = 0; g < 8; ++g) {
    const u32 A = v[2 * g], B = v[2 * g + 1];
    u32 e[4], o[4];
#pragma unroll
    for (int q = 0; q < 4; ++q) {
      const u32 x = (q & 1) ? ((q & 2 ? B : A) >> 16) : ((q & 2 ? B : A) & 0xffffu);
      if (S) {
        const u32 d = (x & 0x7ff8u) - (S_U0 << 3);
        const u32 ad = min(d, 255u * 8u) | ((x >> 13) & 4u);
        e[q] = *(LAS const u32*)(enct + ad);
        o[q] = __builtin_amdgcn_ubfe(x - (e[q] >> 16), 3, e[q]);
      } else {
        const u32 d = (x & 0x7ffcu) - shift;
        const u32 ad = min(d, 511u * 4u);
        e[q] = *(LAS const u32*)(enct + ad);
        o[q] = __builtin_amdgcn_ubfe(d - (e[q] >> 16), 2, e[q]);
      }
    }
    const u32 s01 = e[0] + e[1], s012 = s01 + e[2], W = (s012 + e[3]) & 0xffu;
    G[g] = o[0] | (o[1] << (e[0] & 31)) | (o[2] << (s01 & 31)) | (o[3] << (s012 & 31));
    P[g] = run; run += W; mx = max(mx, W);
    const u32 t02 = __builtin_amdgcn_perm(e[2], e[0], 0x0c0c0501u), t13 = __builtin_amdgcn_perm(e[3], e[1], 0x05010c0cu);
    if (S) F[g] = t02 | t13 | (A & 0x00070007u) | ((B & 0x00070007u) << 8);
    else { const u32 Bm = B & 0x80038003u; F[g] = t02 | t13 | (A & 0x80038003u) | __builtin_amdgcn_alignbit(Bm, Bm, 24); }
  }
  const u32 incl = scan_incl(run);
  const u32 mine = __builtin_amdgcn_readlane(incl, 63) | ((delta & 0xffu) << 16) | (__builtin_amdgcn_ballot_w64(mx > 32) != 0 ? 3u << 30 : 1u << 30);
  const u32 other = pair_swap(wid, sk, mine);
  const u32 x0 = h ? other : mine, x1 = h ? mine : other;
  const u32 nvar = ((x0 & 0xffffu) + (x1 & 0xffffu) + 127) >> 7;
  const bool raw = ((x0 | x1) >> 31) || nvar >= 256 || force_raw;
  if (raw) {
    const __amdgpu_buffer_rsrc_t rd = mkrsrc(dst + h * 4096, 4096);
#pragma unroll
    for (int k = 0; k < 4; ++k)
      __builtin_amdgcn_raw_buffer_store_b128(u32x4{v[4 * k], v[4 * k + 1], v[4 * k + 2], v[4 * k + 3]}, rd, lane * 16 + k * 1024, 0, 17);
  } else {
    const __amdgpu_buffer_rsrc_t rd = mkrsrc(dst + h * 2048, 2048);
#pragma unroll
    for (int k = 0; k < 2; ++k)
      __builtin_amdgcn_raw_buffer_store_b128(u32x4{F[4 * k], F[4 * k + 1], F[4 * k + 2], F[4 * k + 3]}, rd, lane * 16 + k * 1024, 0, 17);
    const u32 base = incl - run + (h ? (x0 & 0xffffu) : 0u);
#pragma unroll
    for (int g = 0; g < 8; ++g) {
      const u32 pos = base + P[g];
      const u64 x = (u64)G[g] << (pos & 31);
      LAS u32* p = var + (pos >> 5);
      __hip_atomic_fetch_or(p, (u32)x, __ATOMIC_RELAXED, __HIP_MEMORY_SCOPE_WORKGROUP);
      __hip_atomic_fetch_or(p + 1, (u32)(x >> 32), __ATOMIC_RELAXED, __HIP_MEMORY_SCOPE_WORKGROUP);
    }
  }
  if (raw) return INFO_RAW;
  pair_swap(wid, sk + 1, 1);
  const __amdgpu_buffer_rsrc_t rv = mkrsrc(dst + 4096, nvar * 16);
#pragma unroll 1
  for (u32 k = h; k * 64 < nvar; k += 2) {
    const u32x4 t = *(LAS const u32x4*)((LAS char*)var + k * 1024 + lane * 16);
    __builtin_amdgcn_raw_buffer_store_b128(t, rv, lane * 16 + k * 1024, 0, 17);
  }
  return nvar | (x0 & 0xff0000u) | ((x1 & 0xff0000u) << 8);
}

DEV u32 and_byte1(u32 x, u32 m) {
  u32 r;
  asm("v_and_b32_sdwa %0, %1, %2 dst_sel:DWORD dst_unused:UNUSED_PAD src0_sel:BYTE_1 src1_sel:DWORD" : "=v"(r) : "v"(x), "s"(m));
  return r;
}
DEV u32 pk_add16(u32 x, u32 y) {
  typedef unsigned short u16x2 __attribute__((ext_vector_type(2)));
  return __builtin_bit_cast(u32, (u16x2)(__builtin_bit_cast(u16x2, x) + __builtin_bit_cast(u16x2, y)));
}
template <bool S, class Mid>
DEV void decode_block(u32 (&v)[32], u32 info, u32 lb, int lane, Mid mid) {
  LAS const char* buf = (LAS const char*)lds(lb);
  if (info & INFO_RAW) {
    mid();
#pragma unroll
    for (int k = 0; k < 8; ++k) {
      const u32x4 t = *(LAS const u32x4*)(buf + k * 1024 + lane * 16);
      v[4 * k] = t[0]; v[4 * k + 1] = t[1]; v[4 * k + 2] = t[2]; v[4 * k + 3] = t[3];
    }
    return;
  }
  LAS const char* dect = (LAS const char*)lds((S ? L_DECS : L_DECU));
  u32 F[16];
#pragma unroll
  for (int k = 0; k < 4; ++k) {
    const u32x4 t = *(LAS const u32x4*)(buf + k * 1024 + lane * 16);
    F[4 * k] = t[0]; F[4 * k + 1] = t[1]; F[4 * k + 2] = t[2]; F[4 * k + 3] = t[3];
  }
  u32 e[16][4], s01[16], s012[16], P[16], run[2] = {0, 0};
#pragma unroll
  for (int g = 0; g < 16; ++g) {
    const u32 m = S ? 0xf8u : 0x7cu;
    e[g][0] = *(LAS const u32*)(dect + (F[g] & m));
    e[g][1] = *(LAS const u32*)(dect + ((F[g] >> 16) & m));
    e[g][2] = *(LAS const u32*)(dect + and_byte1(F[g], m));
    e[g][3] = *(LAS const u32*)(dect + ((F[g] >> 24) & m));
    s01[g] = e[g][0] + e[g][1]; s012[g] = s01[g] + e[g][2];
    P[g] = run[g >> 3]; run[g >> 3] += s012[g] + e[g][3];
  }
  mid();
  const u32 both = __builtin_amdgcn_perm(run[1], run[0], 0x05040100u);
  const u32 incl = scan_incl(both), excl = incl - both;
  const u32 base[2] = {excl, (excl >> 16) + __builtin_amdgcn_readlane(incl, 63)};
  const u32 dsh[2] = {((info >> 16) & 0xffu) * 0x00800080u, (info >> 24) * 0x00800080u};
#pragma unroll
  for (int g = 0; g < 16; ++g) {
    const u32 pos = base[g >> 3] + P[g];
    LAS const u32* p = (LAS const u32*)(buf + 4096) + __builtin_amdgcn_ubfe(pos, 5, 11);
    const u32 win = __builtin_amdgcn_alignbit(p[1], p[0], pos);
    constexpr int SH = S ? 19 : 18;
    const u32 t0 = (__builtin_amdgcn_ubfe(win, 0, e[g][0]) << SH) + e[g][0];
    const u32 t1 = (__builtin_amdgcn_ubfe(win, e[g][0], e[g][1]) << SH) + e[g][1];
    const u32 t2 = (__builtin_amdgcn_ubfe(win, s01[g], e[g][2]) << SH) + e[g][2];
    const u32 t3 = (__builtin_amdgcn_ubfe(win, s012[g], e[g][3]) << SH) + e[g][3];
    u32 pa = __builtin_amdgcn_perm(t1, t0, 0x07060302u), pb = __builtin_amdgcn_perm(t3, t2, 0x07060302u);
    if (!S) { pa = pk_add16(pa, dsh[g >> 3]); pb = pk_add16(pb, dsh[g >> 3]); }
    const u32 m = S ? 0xfff8fff8u : 0x7ffc7ffcu;
    const u32 Fr = __builtin_amdgcn_alignbit(F[g], F[g], 8);
    v[2 * g] = (pa & m) | (F[g] & ~m);
    v[2 * g + 1] = (pb & m) | (Fr & ~m);
  }
}
DEV u32 block_bytes(u32 info) { return (info & INFO_RAW) ? 8192u : 4096u + (info & 0xffu) * 16u; }
DEV u32x4 srd(const char* p, u32 bytes) { return u32x4{(u32)(size_t)p, (u32)((size_t)p >> 32) & 0xffffu, bytes, 0x00020000u}; }
DEV void dma2(u32x4 rs, u32 lb, u32 vo) {
  u32 keep;
  asm volatile("s_nop 4\n\ts_mov_b32 %0, m0\n\ts_mov_b32 m0, %2\n\ts_nop 0\n\t"
               "buffer_load_dwordx4 %3, %1, 0 offen sc0 sc1 lds\n\t"
               "buffer_load_dwordx4 %3, %1, 0 offen offset:1024 sc0 sc1 lds\n\t"
               "s_mov_b32 m0, %0\n\ts_waitcnt vmcnt(%4)"
               : "=&s"(keep) : "s"(rs), "s"(lb), "v"(vo), "i"(2 * KSTEP) : "memory");
}
DEV bool expired(u32* tmo, u32 call, u64 t0) {
  const u64 el = __builtin_amdgcn_s_memrealtime() - t0;
  if (el < 5000) return false;
  if (el > DEADLINE || __hip_atomic_load(tmo, __ATOMIC_RELAXED, __HIP_MEMORY_SCOPE_AGENT) == call) {
    __hip_atomic_store(tmo, call, __ATOMIC_RELAXED, __HIP_MEMORY_SCOPE_AGENT);
    return true;
  }
  return false;
}
DEV void wait_steps(const char* me, int pi, int pj, int lane, u64 need, u32 call, u32* tmo) {
  const u64* fa = (const u64*)(me + OFF_DONE_A) + pi * 64 + lane;
  const u64* fb = (const u64*)(me + OFF_DONE_B) + pj * 64 + lane;
  const u64 t0 = __builtin_amdgcn_s_memrealtime();
  for (;;) {
    const u64 x = __hip_atomic_load(fa, __ATOMIC_RELAXED, __HIP_MEMORY_SCOPE_SYSTEM);
    const u64 y = __hip_atomic_load(fb, __ATOMIC_RELAXED, __HIP_MEMORY_SCOPE_SYSTEM);
    const bool ok = (u32)x >= call && (u32)(x >> 32) >= call && (u32)y >= call && (u32)(y >> 32) >= call;
    if ((__builtin_amdgcn_ballot_w64(ok) & need) == need) return;
    if (rfl(expired(tmo, call, t0))) return;
    __builtin_amdgcn_s_sleep(PSLEEP);
  }
}

DEV float gelu(float y) {
  const float k1 = -2.0f * 0.7978845608028654f * 1.4426950408889634f, k3 = k1 * 0.044715f;
  const float t = y * __builtin_fmaf(y * y, k3, k1);
  return y * __builtin_amdgcn_rcpf(1.0f + __builtin_amdgcn_exp2f(t));
}
DEV u32 pack_bf16(float lo, float hi) {
  return __builtin_bit_cast(u32, __builtin_convertvector(f32x2{lo, hi}, bf2));
}

template <int MI>
DEV void gemm_tile(const Args& a, int tile, int hh, u32 call, int wid, int lane, u64 tstart, int sk) {
  const int rank = a.rank, ri = rank >> 2, ci = rank & 3;
  char* const me = a.ws[rank];
  u32* const tmo = (u32*)(me + OFF_TMO);
  const int pi = k_tiles.t[tile] >> 3, pj = k_tiles.t[tile] & 7;
  if (wid == 0) wait_steps(me, pi, pj, lane, 0xffull << (8 * rank), call, tmo);
  wg_barrier();
  TS(4);
  if (wid >= 4) {
    const int lw = wid - 4;
    const char* ga = me + OFF_AST + (size_t)pi * 64 * IMG + hh * 8192 + lw * (MI * 1024) + lane * 16;
    const char* gb = me + OFF_BST + (size_t)pj * 64 * IMG + lw * 4096 + lane * 16;
    auto issue = [&](int s, int slot) {
      const int st = (8 * rank + s) & 63;
      LAS char* la = lds(L_RING + slot * SLOT + lw * (MI * 1024));
      LAS char* lb = lds(L_RING + slot * SLOT + IMG + lw * 4096);
#pragma unroll
      for (int i = 0; i < MI; ++i)
        __builtin_amdgcn_global_load_lds((const GAS void*)(ga + (size_t)st * IMG + i * 1024), (LAS void*)(la + i * 1024), 16, 0, 17);
#pragma unroll
      for (int i = 0; i < 4; ++i)
        __builtin_amdgcn_global_load_lds((const GAS void*)(gb + (size_t)st * IMG + i * 1024), (LAS void*)(lb + i * 1024), 16, 0, 17);
    };
    for (int s = 0; s < NSLOT; ++s) issue(s, s);
    asm volatile("s_waitcnt vmcnt(%0)" :: "i"(3 * (MI + 4)) : "memory");
    __builtin_amdgcn_s_barrier();
    asm volatile("s_waitcnt vmcnt(%0)" :: "i"(2 * (MI + 4)) : "memory");
    __builtin_amdgcn_s_barrier();
    for (int s = 0; s < 64; ++s) {
      if (s + NSLOT < 64) {
        issue(s + NSLOT, s & (NSLOT - 1));
        asm volatile("s_waitcnt vmcnt(%0)" :: "i"(2 * (MI + 4)) : "memory");
      } else {
        asm volatile("s_waitcnt vmcnt(0)" ::: "memory");
      }
      __builtin_amdgcn_s_barrier();
    }
  } else {
    const int wr = wid >> 1, wc = wid & 1;
    const int row = lane & 15, c = lane >> 4;
    const int aoff = L_RING + wr * (MI * 2048) + 16 * (4 * row + (c ^ (2 * (row >> 3))));
    const int boff = L_RING + IMG + wc * 4096 + 16 * lane;
    f32x4 acc[MI][4];
#pragma unroll
    for (int i = 0; i < MI; ++i)
#pragma unroll
      for (int j = 0; j < 4; ++j) acc[i][j] = f32x4{0, 0, 0, 0};
    bf16x8 a0[2][MI], b0[2][4], a1[2][MI], b1[2][4];
#define RD(A_, B_, slot_) do { LAS const char* pa = lds((slot_) * SLOT + aoff); LAS const char* pb = lds((slot_) * SLOT + boff); \
    _Pragma("unroll") for (int k = 0; k < 2; ++k) { \
      _Pragma("unroll") for (int i = 0; i < MI; ++i) A_[k][i] = *(LAS const bf16x8*)(pa + i * 2048 + k * 1024); \
      _Pragma("unroll") for (int i = 0; i < 4; ++i) B_[k][i] = *(LAS const bf16x8*)(pb + k * 8192 + i * 1024); } } while (0)
#define MM(A_, B_) do { _Pragma("unroll") for (int k = 0; k < 2; ++k) \
    _Pragma("unroll") for (int i = 0; i < MI; ++i) _Pragma("unroll") for (int j = 0; j < 4; ++j) \
      acc[i][j] = __builtin_amdgcn_mfma_f32_16x16x32_bf16(A_[k][i], B_[k][j], acc[i][j], 0, 0, 0); } while (0)
#define SCHEDG do { _Pragma("unroll") for (int q = 0; q < 2 * MI + 8; ++q) { __builtin_amdgcn_sched_group_barrier(0x8, 1, 0); \
    __builtin_amdgcn_sched_group_barrier(0x100, 1, 0); } __builtin_amdgcn_sched_group_barrier(0x8, 6 * MI - 8, 0); } while (0)
    __builtin_amdgcn_s_barrier();
    RD(a0, b0, 0);
    __builtin_amdgcn_s_waitcnt(0xC07F);
    __builtin_amdgcn_sched_barrier(0);
    __builtin_amdgcn_s_barrier();
    for (int s = 0; s < 64; s += 2) {
      RD(a1, b1, (s + 1) & (NSLOT - 1)); MM(a0, b0); SCHEDG;
      __builtin_amdgcn_sched_barrier(0);
      __builtin_amdgcn_s_waitcnt(0xC07F);
      __builtin_amdgcn_s_barrier();
      __builtin_amdgcn_sched_barrier(0);
      RD(a0, b0, (s + 2) & (NSLOT - 1)); MM(a1, b1); SCHEDG;
      __builtin_amdgcn_sched_barrier(0);
      __builtin_amdgcn_s_waitcnt(0xC07F);
      if (s == 8 - NSLOT - 2 && wid == 0) {
        const u32 pos = max(2 * pj + 1, pi < 8 ? 2 * pi + 2 : pi + 9);
        while (__builtin_amdgcn_s_memrealtime() - tstart < EARLIEST + pos * (16 * 42 * TLINE / PPL / 256) - RAWHEAD * 42 * TLINE / PPL / 256) __builtin_amdgcn_s_sleep(8);
        wait_steps(me, pi, pj, lane, ~0ull, call, tmo);
        TS(5);
      }
      __builtin_amdgcn_s_barrier();
      __builtin_amdgcn_sched_barrier(0);
    }
    TS(6);
#pragma unroll
    for (int i = 0; i < MI; ++i)
#pragma unroll
      for (int r = 0; r < 4; ++r)
        *(LAS f32x4*)lds(L_RING + wc * 32768 + (16 * MI * wr + 16 * i + 4 * c + r) * 256 + 16 * row) =
            f32x4{acc[i][0][r], acc[i][1][r], acc[i][2][r], acc[i][3][r]};
  }
  const bool bad = __hip_atomic_load(tmo, __ATOMIC_RELAXED, __HIP_MEMORY_SCOPE_AGENT) == call;
  char* const dws = a.ws[(rank + 1 + lane / MI) & 7];
  zero_var(wid, lane);
  wg_barrier();
  TS(7);
  const bool act = wid < 2 * MI;
  u32 v[16];
  if (act) {
#pragma unroll
    for (int k = 0; k < 4; ++k) {
      const f32x4 y0 = *(LAS const f32x4*)lds(L_RING + wid * 4096 + k * 1024 + lane * 16);
      const f32x4 y1 = *(LAS const f32x4*)lds(L_RING + 32768 + wid * 4096 + k * 1024 + lane * 16);
      v[4 * k] = pack_bf16(gelu(y0[0]), gelu(y0[1])); v[4 * k + 1] = pack_bf16(gelu(y0[2]), gelu(y0[3]));
      v[4 * k + 2] = pack_bf16(gelu(y1[0]), gelu(y1[1])); v[4 * k + 3] = pack_bf16(gelu(y1[2]), gelu(y1[3]));
    }
    if (bad) {
#pragma unroll
      for (int k = 0; k < 16; ++k) v[k] = 0x7fc07fc0u;
    }
    const u32 info = encode_half<true>(v, wid, lane, me + OFF_COUT + (size_t)(tile * 4 + 2 * hh + (wid >> 1)) * SLOTB,
                                       tile >= 128 - RAWTAIL, sk);
    if (lane == 0) *(LAS u32*)lds(L_XCH + wid * 4) = info;
    TS(8);
    drain();
  }
  wg_barrier();
  if (wid == 0 && lane < 7 * MI) {
    const int k = lane & (MI - 1);
    const u32 inf = *(LAS const u32*)lds(L_XCH + k * 8);
    __hip_atomic_store((u64*)(dws + OFF_HOUT + (rank * 128 + tile) * 128 + (2 * hh + k) * 8), ((u64)call << 32) | inf,
                       __ATOMIC_RELAXED, __HIP_MEMORY_SCOPE_SYSTEM);
  }
  TS(9);
  if (act) {
    const __amdgpu_buffer_rsrc_t rd =
        mkrsrc(a.out + (size_t)(2048 * ri + 128 * pi + 64 * hh + 16 * wid) * 8192 + (1024 * ci + 128 * pj) * 2, 15 * 8192 + 256);
#pragma unroll
    for (int k = 0; k < 4; ++k)
      __builtin_amdgcn_raw_buffer_store_b128(u32x4{v[4 * k], v[4 * k + 1], v[4 * k + 2], v[4 * k + 3]}, rd,
                                             (lane >> 4) * 8192 + (lane & 15) * 16 + k * 32768, 0, 17);
  }
}

__global__ void __launch_bounds__(512, 2) fused_kernel(Args a) {
  const int tid = threadIdx.x, lane = tid & 63, wid = rfl(tid >> 6), b = blockIdx.x;
  const int rank = a.rank, ri = rank >> 2, ci = rank & 3;
  char* const me = a.ws[rank];
  u32* const tmo = (u32*)(me + OFF_TMO);
  TS_INIT();
  TS(0);
  const u64 tstart = __builtin_amdgcn_s_memrealtime();
  u32* const cnt = (u32*)(me + OFF_CALL + (b * 8 + wid) * 64);
  u32 call = __hip_atomic_load(cnt, __ATOMIC_RELAXED, __HIP_MEMORY_SCOPE_AGENT) + 1;

  {
    const int slot = (wid >> 1) ^ (is_pull(b) ? 2 : 0), e = slot * GRID + b, h = wid & 1;
    if (slot < 2) __builtin_amdgcn_s_setprio(2);
    int n, sub = e & 15, grp, t; bool isw = false;
    if (e < 768) { n = e / 96; const int r = e - 96 * n; sub = r & 15; isw = r < 64; grp = isw ? r >> 4 : (r - 64) >> 4; t = 32 * n + (isw ? 0 : 16) + sub; }
    else { n = 8 + ((e - 768) >> 5); grp = (e >> 4) & 1; t = 256 + 16 * (n - 8) + sub; }
    const int ks = sub >> 1, hf = sub & 1;
    const int d = isw ? grp + 4 * lane : 4 * grp + lane;
    const bool dok = lane < (isw ? 2 : 4) && (unsigned)d < (unsigned)NR;
    u32 r[16], v[16];
    if (isw) {
      const char* src = a.w + (size_t)(64 * ks + 32 * hf + 8 * (lane >> 4)) * 8192 + (1024 * grp + 128 * n + 8 * (lane & 15) + 4 * h) * 2;
#pragma unroll
      for (int k = 0; k < 8; ++k) { const u32x2 q = *(const u32x2*)(src + k * 8192); r[2 * k] = q[0]; r[2 * k + 1] = q[1]; }
    } else {
      const int rr = lane >> 2;
      const char* src = a.x + (size_t)(2048 * grp + 128 * n + 64 * hf + 32 * h + rr) * 1024 + (64 * ks + 8 * ((lane & 3) ^ (2 * (rr >> 3)))) * 2;
#pragma unroll
      for (int k = 0; k < 4; ++k) {
        const u32x4 q = *(const u32x4*)(src + (k >> 1) * 16384 + (k & 1) * 64);
        r[4 * k] = q[0]; r[4 * k + 1] = q[1]; r[4 * k + 2] = q[2]; r[4 * k + 3] = q[3];
      }
    }
    if (tid < 288) *(LAS u32x4*)lds(tid * 16) = *(const u32x4*)&k_tabs.t[tid * 4];
    char* const dws = a.ws[d & 7];
    zero_var(wid, lane);
    if (tid < 8) *(LAS u32*)lds(L_PQ + tid * 4) = 0;
    if (tid < 64) *(LAS u32*)lds(L_SYNC + tid * 4) = 0;
    wg_barrier();
    call = rfl(call);
    if (lane == 0) __hip_atomic_store(cnt, call, __ATOMIC_RELAXED, __HIP_MEMORY_SCOPE_AGENT);
    if (isw) {
#pragma unroll
      for (int c = 0; c < 4; ++c)
#pragma unroll
        for (int j = 0; j < 4; ++j)
          v[4 * c + j] = __builtin_amdgcn_perm(r[2 * (2 * j + 1) + (c >> 1)], r[2 * (2 * j) + (c >> 1)], (c & 1) ? 0x07060302u : 0x05040100u);
    } else {
#pragma unroll
      for (int k = 0; k < 16; ++k) v[k] = r[k];
    }
    TS(1);
    const bool self = isw ? (grp == ci) : (grp == ri);
    const int stepi = (n * 64 + rank * 8 + ks) * 2 + hf;
    const u32 sto = (u32)(isw ? OFF_BST : OFF_AST) + stepi * 8192 + h * 4096, fo = (u32)(isw ? OFF_DONE_B : OFF_DONE_A) + stepi * 4;
    if (t < RAWHEAD) {
      while (__builtin_amdgcn_s_memrealtime() - tstart < (u64)(PUSH_T0 + t * PUSH_DT)) __builtin_amdgcn_s_sleep(1);
#pragma unroll 1
      for (int q = 0; q < (isw ? 2 : 4); ++q) {
        const __amdgpu_buffer_rsrc_t rd = mkrsrc(a.ws[(isw ? grp + 4 * q : 4 * grp + q) & 7] + sto, 4096);
#pragma unroll
        for (int k = 0; k < 4; ++k)
          __builtin_amdgcn_raw_buffer_store_b128(u32x4{v[4 * k], v[4 * k + 1], v[4 * k + 2], v[4 * k + 3]}, rd, lane * 16 + k * 1024, 0, 17);
      }
      TS(2);
      drain();
      pair_swap(wid, 2, 1);
      if (h == 0 && dok) __hip_atomic_store((u32*)(dws + fo), call, __ATOMIC_RELAXED, __HIP_MEMORY_SCOPE_SYSTEM);
    } else {
      const u32 info = encode_half<false>(v, wid, lane, me + OFF_CIN + (size_t)e * SLOTB, false, 0);
      if (self) {
        const __amdgpu_buffer_rsrc_t rd = mkrsrc(me + sto, 4096);
#pragma unroll
        for (int k = 0; k < 4; ++k)
          __builtin_amdgcn_raw_buffer_store_b128(u32x4{v[4 * k], v[4 * k + 1], v[4 * k + 2], v[4 * k + 3]}, rd, lane * 16 + k * 1024, 0, 17);
      }
      TS(2);
      drain();
      pair_swap(wid, 2, 1);
      if (h == 0) {
        if (self && lane == 0) __hip_atomic_store((u32*)(me + fo), call, __ATOMIC_RELAXED, __HIP_MEMORY_SCOPE_AGENT);
        while (__builtin_amdgcn_s_memrealtime() - tstart < (u64)(HDR_T0 + (((t - RAWHEAD) * HDR_DT8) >> 3))) __builtin_amdgcn_s_sleep(1);
        if (dok && d != rank)
          __hip_atomic_store((u64*)(dws + OFF_HIN + (rank * NIN + t) * 128), ((u64)call << 32) | info, __ATOMIC_RELAXED, __HIP_MEMORY_SCOPE_SYSTEM);
      }
    }
    __builtin_amdgcn_s_setprio(0);
    TS(3);
  }

  const u32 role = k_roles.t[b];
  if ((role & 0xffff) >= R_PULL) {
    if (wid == 4 || wid == 6 || role == R_IDLE) { TS_DUMP(); return; }
    const u32 link = (role - R_PULL) % 7, f = (role - R_PULL) / 7;
    const u32 p = (rank + 1 + link) & 7;
    constexpr u32 Q_SCH = L_PQ, Q_LAND = L_PQ + 4, Q_CUR = L_PQ + 8, Q_DONE = L_PQ + 16, Q_DESC = L_PQ + 32;
    if (wid == 7) {
      u32* const tkp = (u32*)(me + OFF_TKT + (call & 1) * 1024 + link * 128);
      if (f == 0 && lane == 0) __hip_atomic_store((u32*)(me + OFF_TKT + (~call & 1) * 1024 + link * 128), 0u, __ATOMIC_RELAXED, __HIP_MEMORY_SCOPE_AGENT);
      u64 t0 = __builtin_amdgcn_s_memrealtime();
      bool dead = false;
      u32 tk = RAWHEAD + f + lane * PPL, npend = LOOK + 1, nend = 0;
      for (u32 i = 0; nend < 4;) {
        const u32 j = i + lane, cur = lds_get(Q_CUR);
        const u32 want = min(cur + 1 + LOOK - min(i + npend, cur + 1 + LOOK), 4u);
        const u32 dn = *(volatile LAS u32*)lds(Q_DONE + (j & 3) * 4);
        const u32 oi = (u32)OFF_HIN + (p * NIN + tk) * 128, oo = (u32)OFF_HOUT + (p * 128 + ((tk - NIN) >> 2)) * 128 + (tk & 3) * 8;
        const bool room = lane < npend && j < 4 * dn + (j & 3) + NBUF;
        u64 x = 0;
        if (room && tk < NTOT) x = __hip_atomic_load((const u64*)(me + (tk < NIN ? oi : oo)), __ATOMIC_RELAXED, __HIP_MEMORY_SCOPE_SYSTEM);
        u32 got = 0;
        if (want && lane == 0) got = __hip_atomic_fetch_add(tkp, want, __ATOMIC_RELAXED, __HIP_MEMORY_SCOPE_AGENT);
        got = rfl(got) + RAWHEAD + (LOOK + 1) * PPL;
        const bool ok = room && (dead || tk >= NTOT || (u32)(x >> 32) >= call);
        const u32 n = __builtin_ctz(~(u32)__builtin_amdgcn_ballot_w64(ok));
        if (lane < n) *(volatile LAS u32x2*)lds(Q_DESC + (j % NBUF) * 8) = u32x2{tk >= NTOT ? INFO_END : dead ? INFO_BAD : (u32)x, tk};
        nend += __builtin_popcount((u32)__builtin_amdgcn_ballot_w64(lane < n && tk >= NTOT));
        if (lane >= npend) tk = min(got + (lane - npend), NTOT);
        tk = __builtin_amdgcn_ds_bpermute((lane + n) * 4, tk);
        npend += want - n;
        if (n) {
          i += n;
          lds_put(Q_SCH, i);
          t0 = __builtin_amdgcn_s_memrealtime();
        } else if (!want) {
          dead = dead || rfl(expired(tmo, call, t0));
          __builtin_amdgcn_s_sleep(2);
        }
      }
    } else if (wid == 5) {
      __builtin_amdgcn_s_setprio(3);
      const char* const peer = a.ws[p];
      while (__builtin_amdgcn_s_memrealtime() - tstart < PULL_T0) __builtin_amdgcn_s_sleep(1);
      u64 base = __builtin_amdgcn_s_memrealtime();
      u32 due = 0, sch = 0, pub = 0;
      for (u32 i = 0, nend = 0; nend < 4; ++i) {
        lds_put(Q_CUR, i);
        if (i >= sch) {
          sch = lds_get(Q_SCH);
          if (i >= sch) {
            drain();
            lds_put(Q_LAND, pub = i);
            do { __builtin_amdgcn_s_sleep(1); sch = lds_get(Q_SCH); } while (i >= sch);
            const u64 now = __builtin_amdgcn_s_memrealtime(), el = (now - base) * 256;
            due = max(el < due ? due - (u32)el : 0u, f * (16 * TLINE / PPL));
            base = now;
          }
        }
        const u32 info = lds_get(Q_DESC + (i % NBUF) * 8), t = lds_get(Q_DESC + (i % NBUF) * 8 + 4);
        if (info == INFO_END) {
          drain();
          lds_put(Q_LAND, pub = i + 1);
          ++nend;
          continue;
        }
        const u32 e0 = 96 * (t >> 5) + ((t & 16) ? 64 + 16 * ri : 16 * ci), e1 = 768 + 2 * ((t - 256) & ~15u) + 16 * ri;
        const u32 oi = (u32)OFF_CIN + ((t < 256 ? e0 : e1) + (t & 15)) * SLOTB, oo = (u32)OFF_COUT + (t - NIN) * SLOTB;
        const u32 nb = info == INFO_BAD ? 0 : block_bytes(info), lb = L_RING + (i % NBUF) * PBUF;
        const u32x4 rs = srd(peer + (t < NIN ? oi : oo), nb);
        const u32 now = (u32)(__builtin_amdgcn_s_memrealtime() - base) * 256;
        due = max(due, now - min(now, 16 * TLINE));
#pragma unroll 1
        for (u32 o = 0; o < max(nb, 6144u); o += 2048) {
          while ((u32)(__builtin_amdgcn_s_memrealtime() - base) * 256 < due + (o >> 7) * TLINE) __builtin_amdgcn_s_sleep(1);
          dma2(rs, lb + o, lane * 16 + o);
          if (o == (KSTEP - 4) * 2048 && i > pub + 1) lds_put(Q_LAND, pub = i - 1);
        }
        due += ((nb + 127) >> 7) * TLINE;
      }
      TS(30);
    } else {
      const u32 vo0 = lane * 16, voo = (lane >> 4) * 8192 + (lane & 15) * 16;
      u32* pend = nullptr;
      u32 v[32], land = 0;
      auto publish = [&]() {
        if (pend) {
          drain();
          if (lane == 0) __hip_atomic_store(pend, call, __ATOMIC_RELAXED, __HIP_MEMORY_SCOPE_AGENT);
          pend = nullptr;
        }
      };
      for (u32 i = wid;; i += 4) {
        if (i >= land) {
          land = lds_get(Q_LAND);
          if (i >= land) {
            publish();
            for (land = lds_get(Q_LAND); i >= land; land = lds_get(Q_LAND)) __builtin_amdgcn_s_sleep(2);
          }
        }
        const u32 info = lds_get(Q_DESC + (i % NBUF) * 8), t = lds_get(Q_DESC + (i % NBUF) * 8 + 4), lb = L_RING + (i % NBUF) * PBUF;
        if (info == INFO_END) break;
        if (info == INFO_BAD) {
          publish();
#pragma unroll
          for (int k = 0; k < 32; ++k) v[k] = 0x7fc07fc0u;
        } else if (t < NIN) decode_block<false>(v, info, lb, lane, publish);
        else decode_block<true>(v, info, lb, lane, publish);
        lds_put(Q_DONE + wid * 4, (i >> 2) + 1);
        const bool isin = t < NIN, isw = t < 256 && !(t & 16);
        const u32 stepi = (t < 256 ? t >> 5 : 8 + ((t - 256) >> 4)) * 128 + p * 16 + (t & 15);
        const u32 tl = k_tiles.t[((t >> 2) - NIN / 4) & 127], ti = tl >> 3, tj = tl & 7;
        const u32 oi = (u32)(isw ? OFF_BST : OFF_AST) + stepi * 8192;
        const u32 oo = (2048 * (p >> 2) + 128 * ti + 32 * (t & 3)) * 8192 + (1024 * (p & 3) + 128 * tj) * 2;
        pend = isin ? (u32*)(me + (isw ? OFF_DONE_B : OFF_DONE_A)) + stepi : nullptr;
        const __amdgpu_buffer_rsrc_t rd = mkrsrc((isin ? me : a.out) + rfl(isin ? oi : oo), isin ? 8192 : 31 * 8192 + 256);
        const u32 vo = isin ? vo0 : voo, vs = isin ? 1024 : 32768;
#pragma unroll
        for (int k = 0; k < 8; ++k)
          __builtin_amdgcn_raw_buffer_store_b128(u32x4{v[4 * k], v[4 * k + 1], v[4 * k + 2], v[4 * k + 3]}, rd, vo + k * vs, 0, 17);
      }
      publish();
    }
    TS_DUMP();
    return;
  }

  if (ISO) {
#pragma unroll 1
    for (u32 r = role; r; r >>= 16) gemm_tile<2>(a, r & 255, (r >> 9) & 1, call, wid, lane, tstart, r == role ? 3 : 5);
  } else if (role >> 8) gemm_tile<2>(a, role & 255, role >> 9, call, wid, lane, tstart, 3);
  else gemm_tile<4>(a, role, 0, call, wid, lane, tstart, 3);
  TS_DUMP();
}

extern "C" void kernel_launch(int rank, int nranks, ncclComm_t comm,
                              hipStream_t stream,
                              void* const* d_in, const int* in_sizes, int n_in,
                              void* d_out, int out_size,
                              const int* in_dtypes, int out_dtype,
                              void* const* d_out_all,
                              void* const* d_ws_all, size_t ws_bytes) {
  (void)comm; (void)in_dtypes; (void)out_dtype; (void)d_out_all;
  if (nranks != NR || (unsigned)rank >= (unsigned)NR || n_in < 2 || ws_bytes < WS_NEED ||
      in_sizes[0] != 4096 * 512 || in_sizes[1] != 512 * 4096 || out_size != 4096 * 4096) {
    fprintf(stderr, "kernel_launch: unexpected problem shape\n");
    return;
  }
  HIP_CHECK(hipFuncSetAttribute((const void*)fused_kernel, hipFuncAttributeMaxDynamicSharedMemorySize, LDS_BYTES));
  Args a;
  a.x = (const char*)d_in[0];
  a.w = (const char*)d_in[1];
  a.out = (char*)d_out;
  for (int p = 0; p < NR; ++p) a.ws[p] = (char*)d_ws_all[p];
  a.rank = rank;
  a.pad = 0;
  fused_kernel<<<GRID, 512, LDS_BYTES, stream>>>(a);
  HIP_CHECK(hipGetLastError());
}
```

```cpp
#include <hip/hip_runtime.h>
#include <rccl/rccl.h>
#include <cstdio>
#include <cstdint>

#define HIP_CHECK(expr) do { hipError_t _e = (expr); if (_e != hipSuccess) { \
    fprintf(stderr, "HIP error %s at %s:%d\n", hipGetErrorString(_e), __FILE__, __LINE__); \
    return; } } while (0)

typedef unsigned u32;
typedef unsigned long long u64;
typedef unsigned u32x4 __attribute__((ext_vector_type(4)));
typedef unsigned u32x2 __attribute__((ext_vector_type(2)));
typedef short bf16x8 __attribute__((ext_vector_type(8)));
typedef float f32x4 __attribute__((ext_vector_type(4)));
typedef float f32x2 __attribute__((ext_vector_type(2)));
typedef __bf16 bf2 __attribute__((ext_vector_type(2)));
#define GAS __attribute__((address_space(1)))
#define LAS __attribute__((address_space(3)))
#define DEV __device__ __forceinline__

constexpr int NR = 8;
constexpr int GRID = 256;
constexpr int PPL = 8;
constexpr int NPULL = 7 * PPL;
constexpr int LINK_MBS = 60700;
constexpr int KSTEP = 4;
constexpr int LOOK = 8;
constexpr int ENDN = 12;
constexpr int ENDQ = 4;
constexpr int RAWTAIL = 1;
constexpr int NRAW = 12;
constexpr int PUSH_T0 = 120;
constexpr int PUSH_DT = 15;
constexpr int PUSH2_DT = 10;
constexpr int HDR_T0 = 300;
constexpr int LOC_T0 = 550;
constexpr int EARLIEST = 720;
constexpr u32 SLOTB = 9472;
constexpr u32 TLINE = (u32)(PPL * 128.0 * 25600.0 / LINK_MBS + 0.5);
constexpr u32 NIN = 384, NOUT = 512, NTOT = NIN + NOUT;
constexpr u32 NCP = 32 - NRAW, NSEQ = NTOT - 32;
constexpr u64 DEADLINE = 100000000ull;

constexpr size_t OFF_CALL = 0;
constexpr size_t OFF_TMO = 131072;
constexpr size_t OFF_TKT = 131072 + 2048;
constexpr size_t OFF_DONE_A = 131072 + 4096;
constexpr size_t OFF_DONE_B = OFF_DONE_A + 16384;
constexpr size_t OFF_HIN = 262144;
constexpr size_t OFF_HOUT = OFF_HIN + 8 * NIN * 128;
constexpr size_t OFF_CIN = 40u << 20;
constexpr size_t OFF_CPUSH = 2u << 20;
constexpr size_t OFF_COUT = 6u << 20;
constexpr size_t OFF_AST = 16u << 20;
constexpr size_t OFF_BST = 32u << 20;
constexpr size_t WS_NEED = 56u << 20;
static_assert(SLOTB >= 8192 + 128 && SLOTB <= 16384 && SLOTB % 128 == 0, "slot");
static_assert(OFF_HOUT + 8 * 128 * 128 <= OFF_CPUSH, "ctrl region");

constexpr int NSLOT = 4, IMG = 16384, SLOT = 32768;
constexpr int L_ENCU = 0;
constexpr int L_ENCS = 2048;
constexpr int L_DECU = 6144;
constexpr int L_DECS = 6272;
constexpr int L_TABEND = 6528;
constexpr int L_XCH = 6528;
constexpr int L_PQ = 6592;
constexpr u32 NBUF = 16;
constexpr int L_SYNC = 6912;
constexpr int L_RING = 9216;
constexpr int VARSZ = 4096 + 64;
constexpr int PBUF = 8192 + 64;
constexpr int L_VAR = L_RING + 16 * PBUF + 2048;
constexpr int LDS_BYTES = L_VAR + 4 * VARSZ;
static_assert(L_RING + NSLOT * SLOT <= LDS_BYTES && LDS_BYTES <= 163840, "lds");

struct Tab { unsigned short base[64]; unsigned char w[64]; };
constexpr Tab TU = {{0xe20, 0xf20, 0xf60, 0xf70, 0xf80, 0xf88, 0xf90, 0xf98, 0xfa0, 0xfa4, 0xfa8, 0xfb0, 0xfb8, 0xfc0, 0xfc4, 0xfc8,
                     0xfcc, 0xfd0, 0xfd4, 0xfd8, 0xfdc, 0xfe0, 0xfe2, 0xfe4, 0xfe6, 0xfea, 0xfee, 0xff2, 0xff6, 0xffe, 0x1006, 0xe20},
                    {8, 6, 4, 4, 3, 3, 3, 3, 2, 2, 3, 3, 3, 2, 2, 2, 2, 2, 2, 2, 2, 1, 1, 1, 2, 2, 2, 2, 3, 3, 5, 13}};
constexpr Tab TG = {{0xe80, 0xf00, 0xf40, 0xf50, 0xf60, 0xf70, 0xf80, 0xf88, 0xf90, 0xf98, 0xfa0, 0xfa4, 0xfa8, 0xfac, 0xfb0, 0xfb4,
                     0xfb8, 0xfbc, 0xfc0, 0xfc4, 0xfc8, 0xfcc, 0xfd0, 0xfd4, 0xfd8, 0xfdc, 0xfe0, 0xfe2, 0xfe6, 0xfea, 0xfee, 0xff2,
                     0xff6, 0xffe, 0x1006, 0x2e60, 0x2ee0, 0x2f20, 0x2f40, 0x2f48, 0x2f50, 0x2f58, 0x2f60, 0x2f64, 0x2f68, 0x2f6c, 0x2f70, 0x2f74,
                     0x2f78, 0x2f7c, 0x2f7e, 0x2f80, 0x2f81, 0x2f82, 0x2f83, 0x2f84, 0x2f85, 0x2f86, 0x2f87, 0x2f88, 0x2f89, 0x2f8a, 0x2f8b, 0},
                    {7, 6, 4, 4, 4, 4, 3, 3, 3, 3, 2, 2, 2, 2, 2, 2, 2, 2, 2, 2, 2, 2, 2, 2, 2, 2, 1, 2, 2, 2, 2, 2,
                     3, 3, 5, 7, 6, 5, 3, 3, 3, 3, 2, 2, 2, 2, 2, 2, 2, 1, 1, 0, 0, 0, 0, 0, 0, 0, 0, 0, 0, 0, 0, 14}};
constexpr u32 U_U0 = 0xe20, S_U0 = 0xe60;
struct LdsTabs { u32 t[L_TABEND / 4]; };
constexpr LdsTabs make_tabs() {
  LdsTabs r{};
  for (u32 i = 0; i < 512; ++i) {
    u32 eu = 13u | (31u << 10), e0 = 14u | (63u << 10), e1 = e0;
    for (u32 c = 0; c < 63 && i < 511; ++c) {
      if (c < 31 && i - (TU.base[c] - U_U0) < (1u << TU.w[c])) eu = TU.w[c] | (c << 10) | ((TU.base[c] - U_U0) << 18);
      if (i + S_U0 - TG.base[c] < (1u << TG.w[c])) e0 = TG.w[c] | (c << 10) | ((u32)TG.base[c] << 18);
      if (i + S_U0 + 0x2000 - TG.base[c] < (1u << TG.w[c])) e1 = TG.w[c] | (c << 10) | ((u32)TG.base[c] << 18);
    }
    r.t[L_ENCU / 4 + i] = eu; r.t[L_ENCS / 4 + i] = e0; r.t[L_ENCS / 4 + 512 + i] = e1;
  }
  for (u32 c = 0; c < 64; ++c) {
    if (c < 32) r.t[L_DECU / 4 + c] = ((u32)TU.base[c] << 18) | TU.w[c];
    r.t[L_DECS / 4 + c] = ((u32)TG.base[c] << 18) | TG.w[c];
  }
  return r;
}
__device__ const LdsTabs __attribute__((aligned(16))) k_tabs = make_tabs();
struct TileTab { u32 t[128]; };
constexpr TileTab make_tiles() {
  TileTab r{};
  u32 o = 0;
  for (u32 n = 0; n < 8; ++n) {
    for (u32 j = 0; j < n; ++j) r.t[o++] = 8 * n + j;
    for (u32 i = 0; i <= n; ++i) r.t[o++] = 8 * i + n;
  }
  for (; o < 128; ++o) r.t[o] = o;
  return r;
}
__device__ const TileTab k_tiles = make_tiles();
__host__ __device__ constexpr bool is_pull(u32 b) { return !(b & 1) && b >= GRID - 2 * NPULL; }
struct RoleTab { u32 t[128]; };
constexpr RoleTab make_roles() {
  RoleTab r{};
  u32 used[4] = {}, job = 0;
  for (u32 run = 1; run < 16; ++run) {
    u32 c = 0;
    for (u32 k = 1; k < 4; ++k) if (used[k] < used[c]) c = k;
    for (u32 t = 0; t < (run + 1) / 2; ++t) {
      if (used[c] > 30) for (u32 k = 0; k < 4; ++k) if (used[k] < used[c]) c = k;
      for (u32 h = 0; h < 2; ++h, ++job, ++used[c]) {
        const u32 j = (job >> 1) | h << 8 | 0x200;
        r.t[4 * used[c] + c] = j | (j + 64) << 16;
      }
    }
  }
  return r;
}
__device__ const RoleTab k_roles = make_roles();
constexpr u32 INFO_RAW = 1u << 12, INFO_BAD = 1u << 13, INFO_END = 1u << 14;

struct Args {
  const char* x;
  const char* w;
  char* me;
  char* out;
  int rank;
  int pad;
  char* ws[NR];
};

DEV LAS char* lds(u32 off) { return (LAS char*)(size_t)off; }
DEV u32 rfl(u32 v) { return __builtin_amdgcn_readfirstlane(v); }
DEV __amdgpu_buffer_rsrc_t mkrsrc(const void* p, u32 bytes) {
  return __builtin_amdgcn_make_buffer_rsrc((void*)p, 0, bytes, 0x00020000);
}
DEV void drain() { asm volatile("s_waitcnt vmcnt(0)" ::: "memory"); }
DEV void cfence() { asm volatile("" ::: "memory"); }
DEV u32 scan_incl(u32 x) {
  x += __builtin_amdgcn_update_dpp(0u, x, 0x111, 0xf, 0xf, false);
  x += __builtin_amdgcn_update_dpp(0u, x, 0x112, 0xf, 0xf, false);
  x += __builtin_amdgcn_update_dpp(0u, x, 0x114, 0xf, 0xf, false);
  x += __builtin_amdgcn_update_dpp(0u, x, 0x118, 0xf, 0xf, false);
  x += __builtin_amdgcn_update_dpp(0u, x, 0x142, 0xa, 0xf, false);
  x += __builtin_amdgcn_update_dpp(0u, x, 0x143, 0xc, 0xf, false);
  return x;
}
DEV void wg_barrier() {
  __builtin_amdgcn_s_waitcnt(0xC07F);
  __builtin_amdgcn_s_barrier();
  cfence();
}

DEV u32 lds_get(u32 off) { cfence(); const u32 x = rfl(*(volatile LAS u32*)lds(off)); cfence(); return x; }
DEV void lds_put(u32 off, u32 x) { cfence(); *(volatile LAS u32*)lds(off) = x; cfence(); }
DEV u32 pair_swap(int wid, int k, u32 x) {
  lds_put(L_SYNC + wid * 32 + k * 4, x);
  u32 y;
  while ((y = lds_get(L_SYNC + (wid ^ 1) * 32 + k * 4)) == 0) {}
  return y;
}

DEV void zero_var(int wid, int lane) {
  LAS char* var = (LAS char*)lds(L_VAR + (wid >> 1) * VARSZ + (wid & 1) * 2048);
  *(LAS u32x4*)(var + lane * 16) = u32x4{0, 0, 0, 0};
  *(LAS u32x4*)(var + 1024 + lane * 16) = u32x4{0, 0, 0, 0};
  if (lane < 4) *(LAS u32x4*)(var + 2048 + lane * 16) = u32x4{0, 0, 0, 0};
}
DEV void put_fixed(const u32 (&F)[8], const u32 (&v)[16], bool raw, int h, int lane, char* dst) {
  if (raw) {
    const __amdgpu_buffer_rsrc_t rd = mkrsrc(dst + h * 4096, 4096);
#pragma unroll
    for (int k = 0; k < 4; ++k)
      __builtin_amdgcn_raw_buffer_store_b128(u32x4{v[4 * k], v[4 * k + 1], v[4 * k + 2], v[4 * k + 3]}, rd, lane * 16 + k * 1024, 0, 17);
  } else {
    const __amdgpu_buffer_rsrc_t rd = mkrsrc(dst + h * 2048, 2048);
#pragma unroll
    for (int k = 0; k < 2; ++k)
      __builtin_amdgcn_raw_buffer_store_b128(u32x4{F[4 * k], F[4 * k + 1], F[4 * k + 2], F[4 * k + 3]}, rd, lane * 16 + k * 1024, 0, 17);
  }
}
DEV void put_bits(u32 nvar, int wid, int lane, char* dst) {
  const __amdgpu_buffer_rsrc_t rv = mkrsrc(dst + 4096, nvar * 16);
#pragma unroll 1
  for (u32 k = wid & 1; k * 64 < nvar; k += 2) {
    const u32x4 t = *(LAS const u32x4*)lds(L_VAR + (wid >> 1) * VARSZ + k * 1024 + lane * 16);
    __builtin_amdgcn_raw_buffer_store_b128(t, rv, lane * 16 + k * 1024, 0, 17);
  }
}
template <bool S>
DEV u32 encode_half(const u32 (&v)[16], u32 (&F)[8], int wid, int lane, char* dst, bool force_raw, int sk) {
  const int h = wid & 1;
  LAS u32* var = (LAS u32*)lds(L_VAR + (wid >> 1) * VARSZ);
  LAS const char* enct = (LAS const char*)lds((S ? L_ENCS : L_ENCU));
  u32 delta = 0;
  if (!S) {
    u32 s = 0;
#pragma unroll
    for (int k = 0; k < 8; ++k) s += (v[2 * k] >> 7) & 0x00ff00ffu;
    s = (s & 0xffffu) + (s >> 16);
    s = __builtin_amdgcn_readlane(scan_incl(s), 63);
    delta = (u32)(((int)s - 128600 + 512) >> 10);
  }
  u32 shift = (U_U0 << 2) + (delta << 7);
  asm("" : "+s"(shift));
  u32 G[8], P[8], run = 0, mx = 0;
#pragma unroll
  for (int g = 0; g < 8; ++g) {
    const u32 A = v[2 * g], B = v[2 * g + 1];
    u32 e[4], o[4];
#pragma unroll
    for (int q = 0; q < 4; ++q) {
      const u32 x = (q & 1) ? ((q & 2 ? B : A) >> 16) : ((q & 2 ? B : A) & 0xffffu);
      if (S) {
        const u32 d = (x & 0x7ffcu) - (S_U0 << 2);
        const u32 ad = min(d, 511u * 4u) | ((x >> 4) & 0x800u);
        e[q] = *(LAS const u32*)(enct + ad);
        o[q] = __builtin_amdgcn_ubfe(x - (e[q] >> 16), 2, e[q]);
      } else {
        const u32 d = (x & 0x7ffcu) - shift;
        const u32 ad = min(d, 511u * 4u);
        e[q] = *(LAS const u32*)(enct + ad);
        o[q] = __builtin_amdgcn_ubfe(d - (e[q] >> 16), 2, e[q]);
      }
    }
    const u32 s01 = e[0] + e[1], s012 = s01 + e[2], W = (s012 + e[3]) & 0xffu;
    G[g] = o[0] | (o[1] << (e[0] & 31)) | (o[2] << (s01 & 31)) | (o[3] << (s012 & 31));
    P[g] = run; run += W; mx = max(mx, W);
    const u32 t02 = __builtin_amdgcn_perm(e[2], e[0], 0x0c0c0501u), t13 = __builtin_amdgcn_perm(e[3], e[1], 0x05010c0cu);
    if (S) F[g] = t02 | t13 | (A & 0x00030003u) | ((B & 0x00030003u) << 8);
    else { const u32 Bm = B & 0x80038003u; F[g] = t02 | t13 | (A & 0x80038003u) | __builtin_amdgcn_alignbit(Bm, Bm, 24); }
  }
  const u32 incl = scan_incl(run);
  const u32 mine = __builtin_amdgcn_readlane(incl, 63) | ((delta & 0xffu) << 16) | (__builtin_amdgcn_ballot_w64(mx > 32) != 0 ? 3u << 30 : 1u << 30);
  const u32 other = pair_swap(wid, sk, mine);
  const u32 x0 = h ? other : mine, x1 = h ? mine : other;
  const u32 nvar = ((x0 & 0xffffu) + (x1 & 0xffffu) + (S ? 0 : 32) + 127) >> 7;
  const bool raw = ((x0 | x1) >> 31) || nvar >= 256 || force_raw;
  if (dst) put_fixed(F, v, raw, h, lane, dst);
  if (raw) return INFO_RAW;
  const u32 base = (S ? 0 : 32) + incl - run + (h ? (x0 & 0xffffu) : 0u);
  if (!S && h == 0 && lane == 0) *var = x0 & 0xffffu;
#pragma unroll
  for (int g = 0; g < 8; ++g) {
    const u32 pos = base + P[g];
    const u64 x = (u64)G[g] << (pos & 31);
    LAS u32* p = var + (pos >> 5);
    __hip_atomic_fetch_or(p, (u32)x, __ATOMIC_RELAXED, __HIP_MEMORY_SCOPE_WORKGROUP);
    __hip_atomic_fetch_or(p + 1, (u32)(x >> 32), __ATOMIC_RELAXED, __HIP_MEMORY_SCOPE_WORKGROUP);
  }
  pair_swap(wid, sk + 1, 1);
  if (dst) put_bits(nvar, wid, lane, dst);
  return S ? nvar | x0 << 16 : nvar | (x0 & 0xff0000u) | ((x1 & 0xff0000u) << 8);
}

DEV u32 and_byte1(u32 x, u32 m) {
  u32 r;
  asm("v_and_b32_sdwa %0, %1, %2 dst_sel:DWORD dst_unused:UNUSED_PAD src0_sel:BYTE_1 src1_sel:DWORD" : "=v"(r) : "v"(x), "s"(m));
  return r;
}
DEV u32 pk_add16(u32 x, u32 y) {
  typedef unsigned short u16x2 __attribute__((ext_vector_type(2)));
  return __builtin_bit_cast(u32, (u16x2)(__builtin_bit_cast(u16x2, x) + __builtin_bit_cast(u16x2, y)));
}
template <bool S, class Mid>
DEV void decode_half(u32 (&v)[16], u32 info, u32 lb, int h, int lane, Mid mid) {
  LAS const char* buf = (LAS const char*)lds(lb);
  if (info & INFO_RAW) {
    mid();
#pragma unroll
    for (int k = 0; k < 4; ++k) {
      const u32x4 t = *(LAS const u32x4*)(buf + h * 4096 + k * 1024 + lane * 16);
      v[4 * k] = t[0]; v[4 * k + 1] = t[1]; v[4 * k + 2] = t[2]; v[4 * k + 3] = t[3];
    }
    return;
  }
  LAS const char* dect = (LAS const char*)lds((S ? L_DECS : L_DECU));
  u32 F[8];
#pragma unroll
  for (int k = 0; k < 2; ++k) {
    const u32x4 t = *(LAS const u32x4*)(buf + h * 2048 + k * 1024 + lane * 16);
    F[4 * k] = t[0]; F[4 * k + 1] = t[1]; F[4 * k + 2] = t[2]; F[4 * k + 3] = t[3];
  }
  const u32 first = S ? info >> 16 : *(LAS const u32*)(buf + 4096);
  u32 e[8][4], s01[8], s012[8], P[8], run = 0;
#pragma unroll
  for (int g = 0; g < 8; ++g) {
    const u32 m = S ? 0xfcu : 0x7cu;
    e[g][0] = *(LAS const u32*)(dect + (F[g] & m));
    e[g][1] = *(LAS const u32*)(dect + ((F[g] >> 16) & m));
    e[g][2] = *(LAS const u32*)(dect + and_byte1(F[g], m));
    e[g][3] = *(LAS const u32*)(dect + ((F[g] >> 24) & m));
    s01[g] = e[g][0] + e[g][1]; s012[g] = s01[g] + e[g][2];
    P[g] = run; run += s012[g] + e[g][3];
  }
  mid();
  const u32 base = scan_incl(run) - run + (S ? 0 : 32) + (h ? first : 0u);
  const u32 dsh = ((info >> (16 + 8 * h)) & 0xffu) * 0x00800080u;
#pragma unroll
  for (int g = 0; g < 8; ++g) {
    const u32 pos = base + P[g];
    LAS const u32* p = (LAS const u32*)(buf + 4096) + __builtin_amdgcn_ubfe(pos, 5, 11);
    const u32 win = __builtin_amdgcn_alignbit(p[1], p[0], pos);
    const u32 t0 = (__builtin_amdgcn_ubfe(win, 0, e[g][0]) << 18) + e[g][0];
    const u32 t1 = (__builtin_amdgcn_ubfe(win, e[g][0], e[g][1]) << 18) + e[g][1];
    const u32 t2 = (__builtin_amdgcn_ubfe(win, s01[g], e[g][2]) << 18) + e[g][2];
    const u32 t3 = (__builtin_amdgcn_ubfe(win, s012[g], e[g][3]) << 18) + e[g][3];
    u32 pa = __builtin_amdgcn_perm(t1, t0, 0x07060302u), pb = __builtin_amdgcn_perm(t3, t2, 0x07060302u);
    if (!S) { pa = pk_add16(pa, dsh); pb = pk_add16(pb, dsh); }
    const u32 m = S ? 0xfffcfffcu : 0x7ffc7ffcu;
    const u32 Fr = __builtin_amdgcn_alignbit(F[g], F[g], 8);
    v[2 * g] = (pa & m) | (F[g] & ~m);
    v[2 * g + 1] = (pb & m) | (Fr & ~m);
  }
}
DEV u32 block_bytes(u32 info) { return (info & INFO_RAW) ? 8192u : 4096u + (info & 0xffu) * 16u; }
DEV u32x4 srd(const char* p, u32 bytes) { return u32x4{(u32)(size_t)p, (u32)((size_t)p >> 32) & 0xffffu, bytes, 0x00020000u}; }
DEV void dma2(u32x4 rs, u32 lb, u32 vo) {
  u32 keep;
  asm volatile("s_nop 4\n\ts_mov_b32 %0, m0\n\ts_mov_b32 m0, %2\n\ts_nop 0\n\t"
               "buffer_load_dwordx4 %3, %1, 0 offen sc0 sc1 lds\n\t"
               "buffer_load_dwordx4 %3, %1, 0 offen offset:1024 sc0 sc1 lds\n\t"
               "s_mov_b32 m0, %0\n\ts_waitcnt vmcnt(%4)"
               : "=&s"(keep) : "s"(rs), "s"(lb), "v"(vo), "i"(2 * KSTEP) : "memory");
}
DEV bool expired(u32* tmo, u32 call, u64 t0) {
  const u64 el = __builtin_amdgcn_s_memrealtime() - t0;
  if (el < 5000) return false;
  if (el > DEADLINE || __hip_atomic_load(tmo, __ATOMIC_RELAXED, __HIP_MEMORY_SCOPE_AGENT) == call) {
    __hip_atomic_store(tmo, call, __ATOMIC_RELAXED, __HIP_MEMORY_SCOPE_AGENT);
    return true;
  }
  return false;
}
DEV void wait_steps(const char* me, int pi, int pj, int hh, int lane, u64 need, u32 call, u32* tmo) {
  const __amdgpu_buffer_rsrc_t ra = mkrsrc(me + OFF_DONE_A + pi * 1024, 1024), rb = mkrsrc(me + OFF_DONE_B + pj * 1024, 1024);
  const u64 t0 = __builtin_amdgcn_s_memrealtime();
  for (;;) {
    cfence();
    const u32x2 x = __builtin_amdgcn_raw_buffer_load_b64(ra, lane * 16 + hh * 8, 0, 17);
    const u32x4 y = __builtin_amdgcn_raw_buffer_load_b128(rb, lane * 16, 0, 17);
    const bool ok = min(min(x[0], x[1]), min(min(y[0], y[1]), min(y[2], y[3]))) >= call;
    cfence();
    if ((__builtin_amdgcn_ballot_w64(ok) & need) == need) return;
    if (rfl(expired(tmo, call, t0))) return;
    __builtin_amdgcn_s_sleep(8);
  }
}

DEV float gelu(float y) {
  const float k1 = -2.0f * 0.7978845608028654f * 1.4426950408889634f, k3 = k1 * 0.044715f;
  const float t = y * __builtin_fmaf(y * y, k3, k1);
  return y * __builtin_amdgcn_rcpf(1.0f + __builtin_amdgcn_exp2f(t));
}
DEV u32 pack_bf16(float lo, float hi) {
  return __builtin_bit_cast(u32, __builtin_convertvector(f32x2{lo, hi}, bf2));
}

DEV void gemm_tile(const Args& a, int tile, int hh, u32 call, int wid, int lane, u64 tstart, int sk) {
  constexpr int MI = 2;
  const int rank = a.rank, ri = rank >> 2, ci = rank & 3;
  char* const me = a.me;
  u32* const tmo = (u32*)(me + OFF_TMO);
  const int pi = k_tiles.t[tile] >> 3, pj = k_tiles.t[tile] & 7;
  if (wid == 0) wait_steps(me, pi, pj, hh, lane, 0xffull << (8 * rank), call, tmo);
  wg_barrier();
  if (wid >= 4) {
    const int lw = wid - 4;
    const char* ga = me + OFF_AST + (size_t)pi * 64 * IMG + hh * 8192 + lw * (MI * 1024) + lane * 16;
    const char* gb = me + OFF_BST + (size_t)pj * 64 * IMG + lw * 4096 + lane * 16;
    auto issue = [&](int s, int slot) {
      const int st = (8 * rank + s) & 63;
      LAS char* la = lds(L_RING + slot * SLOT + lw * (MI * 1024));
      LAS char* lb = lds(L_RING + slot * SLOT + IMG + lw * 4096);
#pragma unroll
      for (int i = 0; i < MI; ++i)
        __builtin_amdgcn_global_load_lds((const GAS void*)(ga + (size_t)st * IMG + i * 1024), (LAS void*)(la + i * 1024), 16, 0, 17);
#pragma unroll
      for (int i = 0; i < 4; ++i)
        __builtin_amdgcn_global_load_lds((const GAS void*)(gb + (size_t)st * IMG + i * 1024), (LAS void*)(lb + i * 1024), 16, 0, 17);
    };
    for (int s = 0; s < NSLOT; ++s) issue(s, s);
    asm volatile("s_waitcnt vmcnt(%0)" :: "i"(3 * (MI + 4)) : "memory");
    __builtin_amdgcn_s_barrier();
    asm volatile("s_waitcnt vmcnt(%0)" :: "i"(2 * (MI + 4)) : "memory");
    __builtin_amdgcn_s_barrier();
    for (int s = 0; s < 64; ++s) {
      if (s + NSLOT < 64) {
        issue(s + NSLOT, s & (NSLOT - 1));
        asm volatile("s_waitcnt vmcnt(%0)" :: "i"(2 * (MI + 4)) : "memory");
      } else {
        asm volatile("s_waitcnt vmcnt(0)" ::: "memory");
      }
      __builtin_amdgcn_s_barrier();
    }
  } else {
    const int wr = wid >> 1, wc = wid & 1;
    const int row = lane & 15, c = lane >> 4;
    const int aoff = L_RING + wr * (MI * 2048) + 16 * (4 * row + (c ^ (2 * (row >> 3))));
    const int boff = L_RING + IMG + wc * 4096 + 16 * lane;
    f32x4 acc[MI][4];
#pragma unroll
    for (int i = 0; i < MI; ++i)
#pragma unroll
      for (int j = 0; j < 4; ++j) acc[i][j] = f32x4{0, 0, 0, 0};
    bf16x8 a0[2][MI], b0[2][4], a1[2][MI], b1[2][4];
#define RD(A_, B_, slot_) do { LAS const char* pa = lds((slot_) * SLOT + aoff); LAS const char* pb = lds((slot_) * SLOT + boff); \
    _Pragma("unroll") for (int k = 0; k < 2; ++k) { \
      _Pragma("unroll") for (int i = 0; i < MI; ++i) A_[k][i] = *(LAS const bf16x8*)(pa + i * 2048 + k * 1024); \
      _Pragma("unroll") for (int i = 0; i < 4; ++i) B_[k][i] = *(LAS const bf16x8*)(pb + k * 8192 + i * 1024); } } while (0)
#define MM(A_, B_) do { _Pragma("unroll") for (int k = 0; k < 2; ++k) \
    _Pragma("unroll") for (int i = 0; i < MI; ++i) _Pragma("unroll") for (int j = 0; j < 4; ++j) \
      acc[i][j] = __builtin_amdgcn_mfma_f32_16x16x32_bf16(A_[k][i], B_[k][j], acc[i][j], 0, 0, 0); } while (0)
#define SCHEDG do { _Pragma("unroll") for (int q = 0; q < 2 * MI + 8; ++q) { __builtin_amdgcn_sched_group_barrier(0x8, 1, 0); \
    __builtin_amdgcn_sched_group_barrier(0x100, 1, 0); } __builtin_amdgcn_sched_group_barrier(0x8, 6 * MI - 8, 0); } while (0)
    __builtin_amdgcn_s_barrier();
    RD(a0, b0, 0);
    __builtin_amdgcn_s_waitcnt(0xC07F);
    __builtin_amdgcn_sched_barrier(0);
    __builtin_amdgcn_s_barrier();
    for (int s = 0; s < 64; s += 2) {
      RD(a1, b1, (s + 1) & (NSLOT - 1)); MM(a0, b0); SCHEDG;
      __builtin_amdgcn_sched_barrier(0);
      __builtin_amdgcn_s_waitcnt(0xC07F);
      __builtin_amdgcn_s_barrier();
      __builtin_amdgcn_sched_barrier(0);
      RD(a0, b0, (s + 2) & (NSLOT - 1)); MM(a1, b1); SCHEDG;
      __builtin_amdgcn_sched_barrier(0);
      __builtin_amdgcn_s_waitcnt(0xC07F);
      if (s == 8 - NSLOT - 2 && wid == 0) {
        const u32 pos = max(pj ? 2 * pj : 1, pi < 8 ? 2 * pi - 1 : pi + 7);
        while (__builtin_amdgcn_s_memrealtime() - tstart < EARLIEST + pos * (16 * 42 * TLINE / PPL / 256)) __builtin_amdgcn_s_sleep(8);
        wait_steps(me, pi, pj, hh, lane, ~0ull, call, tmo);
      }
      __builtin_amdgcn_s_barrier();
      __builtin_amdgcn_sched_barrier(0);
    }
#pragma unroll
    for (int i = 0; i < MI; ++i)
#pragma unroll
      for (int r = 0; r < 4; ++r)
        *(LAS f32x4*)lds(L_RING + wc * 32768 + (16 * MI * wr + 16 * i + 4 * c + r) * 256 + 16 * row) =
            f32x4{acc[i][0][r], acc[i][1][r], acc[i][2][r], acc[i][3][r]};
  }
  const bool bad = __hip_atomic_load(tmo, __ATOMIC_RELAXED, __HIP_MEMORY_SCOPE_AGENT) == call;
  char* const dws = a.ws[(rank + 1 + lane / MI) & 7];
  zero_var(wid, lane);
  wg_barrier();
  const bool act = wid < 2 * MI;
  u32 v[16];
  if (act) {
#pragma unroll
    for (int k = 0; k < 4; ++k) {
      const u32 yo = L_RING + ((lane >> 3) & 1) * 32768 + wid * 4096 + k * 1024 + (lane >> 4) * 256 + (lane & 7) * 32;
      const f32x4 y0 = *(LAS const f32x4*)lds(yo), y1 = *(LAS const f32x4*)lds(yo + 16);
      v[4 * k] = pack_bf16(gelu(y0[0]), gelu(y0[1])); v[4 * k + 1] = pack_bf16(gelu(y0[2]), gelu(y0[3]));
      v[4 * k + 2] = pack_bf16(gelu(y1[0]), gelu(y1[1])); v[4 * k + 3] = pack_bf16(gelu(y1[2]), gelu(y1[3]));
    }
    if (bad) {
#pragma unroll
      for (int k = 0; k < 16; ++k) v[k] = 0x7fc07fc0u;
    }
    u32 F[8];
    const u32 info = encode_half<true>(v, F, wid, lane, me + OFF_COUT + (size_t)(tile * 4 + 2 * hh + (wid >> 1)) * SLOTB,
                                       tile >= 128 - RAWTAIL, sk);
    if (lane == 0) *(LAS u32*)lds(L_XCH + wid * 4) = info;
    drain();
  }
  wg_barrier();
  if (wid == 0 && lane < 7 * MI) {
    const int k = lane & (MI - 1);
    const u32 inf = *(LAS const u32*)lds(L_XCH + k * 8);
    __hip_atomic_store((u64*)(dws + OFF_HOUT + (rank * 128 + tile) * 128 + (2 * hh + k) * 8), ((u64)call << 32) | inf,
                       __ATOMIC_RELAXED, __HIP_MEMORY_SCOPE_SYSTEM);
  }
  if (act) {
    const __amdgpu_buffer_rsrc_t rd =
        mkrsrc(a.out + (size_t)(2048 * ri + 128 * pi + 64 * hh + 16 * wid) * 8192 + (1024 * ci + 128 * pj) * 2, 15 * 8192 + 256);
#pragma unroll
    for (int k = 0; k < 4; ++k)
      __builtin_amdgcn_raw_buffer_store_b128(u32x4{v[4 * k], v[4 * k + 1], v[4 * k + 2], v[4 * k + 3]}, rd,
                                             (lane >> 4) * 8192 + (lane & 15) * 16 + k * 32768, 0, 17);
  }
}

__global__ void __launch_bounds__(512, 2) fused_kernel(Args a) {
  const int tid = threadIdx.x, lane = tid & 63, wid = rfl(tid >> 6), b = blockIdx.x;
  const int rank = a.rank, ri = rank >> 2, ci = rank & 3;
  char* const me = a.me;
  u32* const tmo = (u32*)(me + OFF_TMO);
  const u64 tstart = __builtin_amdgcn_s_memrealtime();
  asm volatile("" :: "s"(a.x), "s"(a.w), "s"(rank));
  if (tid < L_TABEND / 16)
    __builtin_amdgcn_global_load_lds((const GAS void*)&k_tabs.t[tid * 4], (LAS void*)lds(wid * 1024), 16, 0, 0);
  u32* const cnt = (u32*)(me + OFF_CALL + (b * 8 + wid) * 64);
  u32 call = __hip_atomic_load(cnt, __ATOMIC_RELAXED, __HIP_MEMORY_SCOPE_AGENT) + 1;

  {
    const int slot = (wid >> 1) ^ (is_pull(b) ? 2 : 0), h = wid & 1;
    const int e = slot * GRID + b;
    if (slot < 2) __builtin_amdgcn_s_setprio(2);
    int n, sub = e & 15, grp, t; bool isw = false;
    if (e < 768) { n = e / 96; const int r = e - 96 * n; sub = r & 15; isw = r >= 32; grp = isw ? (r - 32) >> 4 : r >> 4; t = 32 * n + (isw ? 16 : 0) + sub; }
    else { n = 8 + ((e - 768) >> 5); grp = (e >> 4) & 1; t = 256 + 16 * (n - 8) + sub; }
    const int ks = sub >> 1, hf = sub & 1;
    const int d = isw ? grp + 4 * lane : 4 * grp + lane;
    const bool dok = lane < (isw ? 2 : 4) && (unsigned)d < (unsigned)NR;
    char* const dws = a.ws[d & 7];
    u32 r[16], v[16];
    if (isw) {
      const char* src = a.w + (size_t)(64 * ks + 32 * hf + 8 * (lane >> 4)) * 8192 + (1024 * grp + 128 * n + 64 * h + 4 * (lane & 15)) * 2;
#pragma unroll
      for (int k = 0; k < 8; ++k) { const u32x2 q = *(const u32x2*)(src + k * 8192); r[2 * k] = q[0]; r[2 * k + 1] = q[1]; }
      asm volatile("s_waitcnt vmcnt(8)" ::: "memory");
    } else {
      const int rr = lane >> 2;
      const char* src = a.x + (size_t)(2048 * grp + 128 * n + 64 * hf + 32 * h + rr) * 1024 + (64 * ks + 8 * ((lane & 3) ^ (2 * (rr >> 3)))) * 2;
#pragma unroll
      for (int k = 0; k < 4; ++k) {
        const u32x4 q = *(const u32x4*)(src + (k >> 1) * 16384 + (k & 1) * 64);
        r[4 * k] = q[0]; r[4 * k + 1] = q[1]; r[4 * k + 2] = q[2]; r[4 * k + 3] = q[3];
      }
      asm volatile("s_waitcnt vmcnt(4)" ::: "memory");
    }
    zero_var(wid, lane);
    if (tid < 8) *(LAS u32*)lds(L_PQ + tid * 4) = 0;
    if (tid < 64) *(LAS u32*)lds(L_SYNC + tid * 4) = 0;
    wg_barrier();
    call = rfl(call);
    if (lane == 0) __hip_atomic_store(cnt, call, __ATOMIC_RELAXED, __HIP_MEMORY_SCOPE_AGENT);
    if (isw) {
#pragma unroll
      for (int c = 0; c < 4; ++c)
#pragma unroll
        for (int j = 0; j < 4; ++j)
          v[4 * c + j] = __builtin_amdgcn_perm(r[2 * (2 * j + 1) + (c >> 1)], r[2 * (2 * j) + (c >> 1)], (c & 1) ? 0x07060302u : 0x05040100u);
    } else {
#pragma unroll
      for (int k = 0; k < 16; ++k) v[k] = r[k];
    }
    const bool self = isw ? (grp == ci) : (grp == ri);
    const int stepi = (n * 64 + rank * 8 + ks) * 2 + hf;
    const u32 sto = (u32)(isw ? OFF_BST : OFF_AST) + stepi * 8192 + h * 4096, fo = (u32)(isw ? OFF_DONE_B : OFF_DONE_A) + stepi * 8 + h * 4;
    auto consumer = [&](int q) {
      const u32 lo = __builtin_amdgcn_readlane((u32)(size_t)dws, q), hi = __builtin_amdgcn_readlane((u32)((size_t)dws >> 32), q);
      return (char*)(((size_t)hi << 32) | lo);
    };
    const int pk = !isw && n < 2 ? 16 * n + sub : 32;
    u32 F[8];
    if (pk < (int)NRAW) {
      const int q0 = (NRAW - pk + 2) / 3;
#pragma unroll 1
      for (int k = 0; k < 4; ++k) {
        const int q = q0 < 4 ? (k + q0) & 3 : k, step = (pk + 3 * q) % NRAW;
        while (__builtin_amdgcn_s_memrealtime() - tstart < (u64)(PUSH_T0 + step * PUSH_DT)) __builtin_amdgcn_s_sleep(1);
        put_fixed(F, v, true, 0, lane, consumer(q) + sto);
      }
      drain();
      if (dok) __hip_atomic_store((u32*)(dws + fo), call, __ATOMIC_RELAXED, __HIP_MEMORY_SCOPE_SYSTEM);
    } else {
      const u32 info = encode_half<false>(v, F, wid, lane, pk < 32 ? nullptr : me + OFF_CIN + (size_t)e * SLOTB, false, 0);
      if (self) put_fixed(F, v, true, 0, lane, me + sto);
      if (pk < 32) {
        const int c = pk - NRAW, q0 = (NCP - c + 4) / 5;
#pragma unroll 1
        for (int k = 0; k < 4; ++k) {
          const int q = q0 < 4 ? (k + q0) & 3 : k, step = (c + 5 * q) % NCP;
          if (self && q == ci) continue;
          while (__builtin_amdgcn_s_memrealtime() - tstart < (u64)(PUSH_T0 + NRAW * PUSH_DT + step * PUSH2_DT)) __builtin_amdgcn_s_sleep(1);
          char* const dq = consumer(q) + OFF_CPUSH + (rank * 32 + pk) * SLOTB;
          put_fixed(F, v, info & INFO_RAW, h, lane, dq);
          if (!(info & INFO_RAW)) put_bits(info & 0xffu, wid, lane, dq);
        }
      }
      drain();
      if (self && lane == 0) __hip_atomic_store((u32*)(me + fo), call, __ATOMIC_RELAXED, __HIP_MEMORY_SCOPE_AGENT);
      pair_swap(wid, 2, 1);
      if (h == 0) {
        while (__builtin_amdgcn_s_memrealtime() - tstart < (u64)(HDR_T0 + max(t - (t < 32 ? 16 : 32), 0))) __builtin_amdgcn_s_sleep(1);
        if (dok && d != rank)
          __hip_atomic_store((u64*)(dws + OFF_HIN + (rank * NIN + t) * 128), ((u64)call << 32) | info, __ATOMIC_RELAXED, __HIP_MEMORY_SCOPE_SYSTEM);
      }
    }
    __builtin_amdgcn_s_setprio(0);
  }

  if (!(b & 1)) {
    if (!is_pull(b)) {
      static_assert(7 * NCP <= 4 * (128 - NPULL), "one wave pair per pushed block");
      const u32 idx = (wid >> 1) * (128 - NPULL) + (b >> 1), h = wid & 1;
      if (idx < 7 * NCP) {
        const u32 p = (rank + 1 + idx / NCP) & 7, pk = NRAW + idx % NCP, t = pk < 16 ? pk : pk + 16, lb = L_RING + (wid >> 1) * PBUF;
        while (__builtin_amdgcn_s_memrealtime() - tstart < LOC_T0) __builtin_amdgcn_s_sleep(4);
        const u64 t0 = __builtin_amdgcn_s_memrealtime();
        u32 info = INFO_BAD;
        for (;;) {
          const u64 x = __hip_atomic_load((const u64*)(me + OFF_HIN + (p * NIN + t) * 128), __ATOMIC_RELAXED, __HIP_MEMORY_SCOPE_SYSTEM);
          if (rfl((u32)(x >> 32)) >= call) { info = rfl((u32)x); break; }
          if (rfl(expired(tmo, call, t0))) break;
          __builtin_amdgcn_s_sleep(8);
        }
        if (pair_swap(wid, 3, info | 1u << 15) == (INFO_BAD | 1u << 15)) info = INFO_BAD;
        u32 v[16];
        if (info != INFO_BAD) {
          const __amdgpu_buffer_rsrc_t rs = mkrsrc(me + OFF_CPUSH + (p * 32 + pk) * SLOTB, block_bytes(info));
#pragma unroll
          for (int k = 0; k < 4; ++k)
            __builtin_amdgcn_raw_ptr_buffer_load_lds(rs, (LAS void*)lds(lb + (2 * k + h) * 1024), 16, (2 * k + h) * 1024 + lane * 16, 0, 0, 17);
          drain();
        }
        pair_swap(wid, 4, 1);
        if (info != INFO_BAD) decode_half<false>(v, info, lb, h, lane, []() {});
        else {
#pragma unroll
          for (int k = 0; k < 16; ++k) v[k] = 0x7fc07fc0u;
        }
        const u32 stepi = (pk >> 4) * 128 + p * 16 + (pk & 15);
        u32 F[8];
        put_fixed(F, v, true, h, lane, me + OFF_AST + stepi * 8192);
        drain();
        if (lane == 0) __hip_atomic_store((u32*)(me + OFF_DONE_A) + stepi * 2 + h, call, __ATOMIC_RELAXED, __HIP_MEMORY_SCOPE_AGENT);
      }
      return;
    }
    if (wid == 4 || wid == 6) return;
    const u32 q = (b >> 1) - (128 - NPULL), link = q % 7, f = q / 7;
    const u32 p = (rank + 1 + link) & 7;
    const char* const peer = a.ws[p];
    constexpr u32 Q_SCH = L_PQ, Q_LAND = L_PQ + 4, Q_CUR = L_PQ + 8, Q_DONE = L_PQ + 16, Q_DESC = L_PQ + 32;
    if (wid == 7) {
      u32* const tkp = (u32*)(me + OFF_TKT + (call & 1) * 1024 + link * 128);
      if (f == 0 && lane == 0) __hip_atomic_store((u32*)(me + OFF_TKT + (~call & 1) * 1024 + link * 128), 0u, __ATOMIC_RELAXED, __HIP_MEMORY_SCOPE_AGENT);
      u64 t0 = __builtin_amdgcn_s_memrealtime();
      bool dead = false;
      u32 tu = f + lane * PPL, npend = LOOK + 1, nend = 0;
      for (u32 i = 0; nend < 2;) {
        const u32 j = i + lane, cur = lds_get(Q_CUR), tk = tu + (tu < 16 ? 16 : 32);
        const u32 want = min(cur + 1 + LOOK - min(i + npend, cur + 1 + LOOK), 4u);
        const u32x2 dn = *(volatile LAS u32x2*)lds(Q_DONE + (j & 1) * 8);
        const u32 e0 = 96 * (tk >> 5) + ((tk & 16) ? 32 + 16 * ci : 16 * ri), e1 = 768 + 2 * ((tk - 256) & ~15u) + 16 * ri;
        const u32 so = tk < NIN ? (u32)OFF_CIN + ((tk < 256 ? e0 : e1) + (tk & 15)) * SLOTB : (u32)OFF_COUT + (tk - NIN) * SLOTB;
        const u32 oi = (u32)OFF_HIN + (p * NIN + tk) * 128, oo = (u32)OFF_HOUT + (p * 128 + ((tk - NIN) >> 2)) * 128 + (tk & 3) * 8;
        const char* const hp = me + (tk < NIN ? oi : oo);
        const bool room = lane < npend && j < 2 * min(dn[0], dn[1]) + (j & 1) + NBUF;
        u64 x = 0;
        if (room && tk < NTOT) x = __hip_atomic_load((const u64*)hp, __ATOMIC_RELAXED, __HIP_MEMORY_SCOPE_SYSTEM);
        u32 got = 0;
        if (want && lane == 0) got = __hip_atomic_fetch_add(tkp, want, __ATOMIC_RELAXED, __HIP_MEMORY_SCOPE_AGENT);
        got = rfl(got) + (LOOK + 1) * PPL;
        const bool ok = room && (dead || tk >= NTOT || (u32)(x >> 32) >= call);
        const u32 n = __builtin_ctz(~(u32)__builtin_amdgcn_ballot_w64(ok));
        const u32 nb = dead ? 0 : block_bytes((u32)x), ns = nb > 6144 ? 4 : 3;
        const u32 dt = ((nb + 127) >> 7) * (NTOT - tk <= ENDN ? TLINE * ENDQ / 8 : TLINE) / ns;
        if (lane < n) *(volatile LAS u32x4*)lds(Q_DESC + (j % NBUF) * 16) =
            u32x4{tk >= NTOT ? INFO_END : dead ? INFO_BAD : (u32)x, tk, tk >= NTOT ? ~0u : so, nb | dt << 14};
        nend += __builtin_popcount((u32)__builtin_amdgcn_ballot_w64(lane < n && tk >= NTOT));
        if (lane >= npend) tu = min(got + (lane - npend), NSEQ);
        tu = __builtin_amdgcn_ds_bpermute((lane + n) * 4, tu);
        npend += want - n;
        if (n) {
          i += n;
          lds_put(Q_SCH, i);
          t0 = __builtin_amdgcn_s_memrealtime();
        } else if (!want) {
          dead = dead || rfl(expired(tmo, call, t0));
          __builtin_amdgcn_s_sleep(8);
        }
      }
    } else if (wid == 5) {
      __builtin_amdgcn_s_setprio(3);
      __builtin_amdgcn_s_waitcnt(0x0F70);
      u64 base = __builtin_amdgcn_s_memrealtime();
      u32 due = 0, sch = 0, pub = 0;
      for (u32 i = 0, nend = 0; nend < 2; ++i) {
        lds_put(Q_CUR, i);
        if (i >= sch) {
          sch = lds_get(Q_SCH);
          if (i >= sch) {
            drain();
            lds_put(Q_LAND, pub = i);
            do { __builtin_amdgcn_s_sleep(1); sch = lds_get(Q_SCH); } while (i >= sch);
            const u64 now = __builtin_amdgcn_s_memrealtime(), el = (now - base) * 256;
            due = max(el < due ? due - (u32)el : 0u, i ? f * (16 * TLINE / PPL) : 0u);
            base = now;
          }
        }
        cfence();
        const u32x2 ds = *(volatile LAS u32x2*)lds(Q_DESC + (i % NBUF) * 16 + 8);
        const u32 so = rfl(ds[0]), nb = rfl(ds[1]) & 0x3fffu, dt = rfl(ds[1]) >> 14;
        if (so == ~0u) {
          drain();
          lds_put(Q_LAND, pub = i + 1);
          ++nend;
          continue;
        }
        const u32 lb = L_RING + (i % NBUF) * PBUF;
        const u32x4 rs = srd(peer + so, nb);
#pragma unroll 1
        for (u32 o = 0; o < max(nb, 6144u); o += 2048) {
          u32 now;
          while ((now = (u32)(__builtin_amdgcn_s_memrealtime() - base) * 256) < due) __builtin_amdgcn_s_sleep(1);
          dma2(rs, lb + o, lane * 16 + o);
          due = max(due + dt, now);
          if (o == (KSTEP - 4) * 2048 && i > pub + 1) lds_put(Q_LAND, pub = i - 1);
        }
      }
    } else {
      const u32 vo0 = lane * 16, voo = (lane >> 4) * 8192 + (lane & 15) * 16, h = wid & 1;
      u32* pend = nullptr;
      u32 v[16], land = 0;
      auto publish = [&]() {
        if (pend) {
          drain();
          if (lane == 0) __hip_atomic_store(pend, call, __ATOMIC_RELAXED, __HIP_MEMORY_SCOPE_AGENT);
          pend = nullptr;
        }
      };
      for (u32 i = wid >> 1;; i += 2) {
        if (i >= land) {
          land = lds_get(Q_LAND);
          if (i >= land) {
            publish();
            for (land = lds_get(Q_LAND); i >= land; land = lds_get(Q_LAND)) __builtin_amdgcn_s_sleep(1);
          }
        }
        cfence();
        const u32x2 ds = *(volatile LAS u32x2*)lds(Q_DESC + (i % NBUF) * 16);
        const u32 info = rfl(ds[0]), t = rfl(ds[1]), lb = L_RING + (i % NBUF) * PBUF;
        if (info == INFO_END) break;
        if (info == INFO_BAD) {
          publish();
#pragma unroll
          for (int k = 0; k < 16; ++k) v[k] = 0x7fc07fc0u;
        } else if (t < NIN) decode_half<false>(v, info, lb, h, lane, publish);
        else decode_half<true>(v, info, lb, h, lane, publish);
        lds_put(Q_DONE + wid * 4, (i >> 1) + 1);
        const bool isin = t < NIN, isw = t < 256 && (t & 16);
        const u32 stepi = (t < 256 ? t >> 5 : 8 + ((t - 256) >> 4)) * 128 + p * 16 + (t & 15);
        const u32 tl = k_tiles.t[((t >> 2) - NIN / 4) & 127], ti = tl >> 3, tj = tl & 7;
        const u32 oi = (u32)(isw ? OFF_BST : OFF_AST) + stepi * 8192 + h * 4096;
        const u32 oo = (2048 * (p >> 2) + 128 * ti + 32 * (t & 3) + 16 * h) * 8192 + (1024 * (p & 3) + 128 * tj) * 2;
        pend = isin ? (u32*)(me + (isw ? OFF_DONE_B : OFF_DONE_A)) + stepi * 2 + h : nullptr;
        const __amdgpu_buffer_rsrc_t rd = mkrsrc((isin ? me : a.out) + rfl(isin ? oi : oo), isin ? 4096 : 15 * 8192 + 256);
        const u32 vo = isin ? vo0 : voo, vs = isin ? 1024 : 32768;
#pragma unroll
        for (int k = 0; k < 4; ++k)
          __builtin_amdgcn_raw_buffer_store_b128(u32x4{v[4 * k], v[4 * k + 1], v[4 * k + 2], v[4 * k + 3]}, rd, vo + k * vs, 0, 17);
      }
      publish();
    }
    return;
  }

  const u32 role = k_roles.t[b >> 1];
#pragma unroll 1
  for (u32 r = role; r; r >>= 16) gemm_tile(a, r & 255, (r >> 8) & 1, call, wid, lane, tstart, r == role ? 3 : 5);
}

extern "C" void kernel_launch(int rank, int nranks, ncclComm_t comm,
                              hipStream_t stream,
                              void* const* d_in, const int* in_sizes, int n_in,
                              void* d_out, int out_size,
                              const int* in_dtypes, int out_dtype,
                              void* const* d_out_all,
                              void* const* d_ws_all, size_t ws_bytes) {
  (void)comm; (void)in_dtypes; (void)out_dtype; (void)d_out_all;
  if (nranks != NR || (unsigned)rank >= (unsigned)NR || n_in < 2 || ws_bytes < WS_NEED ||
      in_sizes[0] != 4096 * 512 || in_sizes[1] != 512 * 4096 || out_size != 4096 * 4096) {
    fprintf(stderr, "kernel_launch: unexpected problem shape\n");
    return;
  }
  HIP_CHECK(hipFuncSetAttribute((const void*)fused_kernel, hipFuncAttributeMaxDynamicSharedMemorySize, LDS_BYTES));
  Args a;
  a.x = (const char*)d_in[0];
  a.w = (const char*)d_in[1];
  a.out = (char*)d_out;
  a.me = (char*)d_ws_all[rank];
  for (int p = 0; p < NR; ++p) a.ws[p] = (char*)d_ws_all[p];
  a.rank = rank;
  a.pad = 0;
  fused_kernel<<<GRID, 512, LDS_BYTES, stream>>>(a);
  HIP_CHECK(hipGetLastError());
}
```
